```python
import numpy as np

kernel_name = "gpu_a2a_gemm_m4096_k4096_n16384_f32_reluqe4m3"


ABSMAX_RTOL = 0.09

COMPUTE_DTYPE = "float32"
EPILOGUE = "relu_quant_e4m3"
IN_QUANT = None
OUT_QUANT = "e4m3"
IN_DTYPES = {"x": "float32", "w_mat": "float32"}
WORLD_SIZE = 8

def setup_inputs():

    x = np.random.randn(4096, 4096).astype(np.float32)
    w_mat = (np.random.randn(4096, 16384) / np.sqrt(4096)).astype(np.float32)
    return {"x": x, "w_mat": w_mat}

def _snap_fp8(x, mant_bits, max_abs, min_normal):
    a = np.abs(x.astype(np.float64))
    _m, _e = np.frexp(a)
    q = np.where(a >= min_normal, np.ldexp(1.0, _e - 1 - mant_bits),
                 min_normal * 2.0 ** -mant_bits)
    snapped = np.minimum(np.round(a / q) * q, max_abs)
    return (np.sign(x) * snapped).astype(np.float32)
def _quant_dequant(y, amax):
    scale = amax / 448.0
    q = _snap_fp8(y / scale, 3, 448.0, 2.0 ** -6)
    return (q * scale).astype(np.float32)
def _epilogue(y):
    y = np.maximum(y, 0.0)
    return _quant_dequant(y, np.abs(y).max())

def reference(x, w_mat):
    y = (x @ w_mat).astype(np.float32)
    return _epilogue(y)
```

```cpp
#include <hip/hip_runtime.h>
#include <rccl/rccl.h>
#include <cstdio>
#include <cstdint>

#define HIP_CHECK(expr) do { hipError_t _e = (expr); if (_e != hipSuccess) { \
    fprintf(stderr, "HIP error %s at %s:%d\n", hipGetErrorString(_e), __FILE__, __LINE__); \
    return; } } while (0)

typedef unsigned u32;
typedef unsigned long long u64;
typedef unsigned u32x4 __attribute__((ext_vector_type(4)));
typedef float f32x2 __attribute__((ext_vector_type(2)));
typedef float f32x4 __attribute__((ext_vector_type(4)));
typedef __bf16 bf16x2 __attribute__((ext_vector_type(2)));
typedef __bf16 bf16x8 __attribute__((ext_vector_type(8)));
typedef __attribute__((address_space(1))) u32 gu32;
typedef __attribute__((address_space(1))) u64 gu64;
typedef __attribute__((address_space(3))) char lchar;

constexpr int NR = 8;
constexpr int MLOC = 512;
constexpr int KTOT = 4096;
constexpr int NTOT = 16384;
constexpr int KC = 512;
constexpr int NWG = 256;
constexpr int NTHR = 512;
constexpr int NST = 64;
constexpr int BD = 4;

constexpr size_t WS_A = 0;
constexpr u32 A_BYTES = (u32)MLOC * KTOT * 2;
constexpr size_t WS_FLAG = A_BYTES;
constexpr size_t WS_GRAN = WS_FLAG + 65536;
constexpr size_t WS_CTR = WS_GRAN + 65536;
constexpr size_t WS_TMO = WS_CTR + 65536;
constexpr size_t WS_END = WS_TMO + 4096;
constexpr u32 W_BYTES = (u32)KTOT * NTOT * 4;

constexpr u32 LDS_A = 0;
constexpr u32 LDS_B = 4 * 32768;
constexpr u32 LDS_BYTES = LDS_B + 2 * 16384;

constexpr u64 DEADLINE_TICKS = 50000000ull;
constexpr u32 OOB = 0x80000000u;

#define S_FLAGB 12
#define WNT ""

struct Params {
    const float* x;
    const float* w;
    unsigned char* out;
    char* ws[NR];
    int rank;
    int pad;
};

__device__ __forceinline__ int kstage_of(int s, int rank) {
    if (s < 8) return rank * 8 + s;
    if (s < 22) { int t = s - 8; return ((rank + 1 + (t >> 1)) & 7) * 8 + (t & 1); }
    int t = s - 22; int q = t / 6;
    return ((rank + 1 + q) & 7) * 8 + 2 + (t - q * 6);
}

__device__ __forceinline__ u32 pack_bf16(u32 a, u32 b) {
    f32x2 f = {__builtin_bit_cast(float, a), __builtin_bit_cast(float, b)};
    return __builtin_bit_cast(u32, __builtin_convertvector(f, bf16x2));
}

__device__ __forceinline__ u32x4 make_srd(const void* p, u32 bytes) {
    u64 a = (u64)p;
    u32x4 r;
    r[0] = __builtin_amdgcn_readfirstlane((u32)a);
    r[1] = __builtin_amdgcn_readfirstlane((u32)(a >> 32)) & 0xffffu;
    r[2] = bytes;
    r[3] = 0x00020000u;
    return r;
}

#define BARRIER() asm volatile("s_waitcnt lgkmcnt(0)\n\ts_barrier" ::: "memory")

__device__ __forceinline__ void store16_sys(char* p, u32x4 v) {
    asm volatile("global_store_dwordx4 %0, %1, off sc0 sc1\n\ts_nop 1" :: "v"(p), "v"(v) : "memory");
}

__device__ __forceinline__ bool poll_flags(const char* p, bool active, u32 call, gu32* tmo) {
    u64 t0 = __builtin_amdgcn_s_memrealtime();
    for (;;) {
        u32x4 v;
        asm volatile("global_load_dwordx4 %0, %1, off sc0 sc1\n\ts_waitcnt vmcnt(0)"
                     : "=&v"(v) : "v"(p) : "memory");
        bool ok = !active || (v[0] >= call && v[1] >= call && v[2] >= call && v[3] >= call);
        if (__all(ok)) return true;
        u32 tv = __hip_atomic_load(tmo, __ATOMIC_RELAXED, __HIP_MEMORY_SCOPE_AGENT);
        if (__builtin_amdgcn_readfirstlane(tv) == call ||
            __builtin_amdgcn_s_memrealtime() - t0 > DEADLINE_TICKS) {
            __hip_atomic_store(tmo, call, __ATOMIC_RELAXED, __HIP_MEMORY_SCOPE_AGENT);
            return false;
        }
        __builtin_amdgcn_s_sleep(2);
    }
}

__device__ __forceinline__ float wave_max(float m) {
    m = fmaxf(m, __shfl_xor(m, 32));
    m = fmaxf(m, __shfl_xor(m, 16));
    m = fmaxf(m, __shfl_xor(m, 8));
    m = fmaxf(m, __shfl_xor(m, 4));
    m = fmaxf(m, __shfl_xor(m, 2));
    m = fmaxf(m, __shfl_xor(m, 1));
    return m;
}

__global__ __launch_bounds__(NTHR, 2) void a2a_gemm_kernel(Params P) {
    extern __shared__ __attribute__((aligned(16))) char smem_g[];
    lchar* smem = (lchar*)smem_g;
    const u32 lds0 = (u32)(size_t)smem;

    const int b = blockIdx.x;
    const int rank = P.rank & (NR - 1);
    const int wave = __builtin_amdgcn_readfirstlane(threadIdx.x >> 6);
    const int lane = threadIdx.x & 63;
    const int mh = (b >> 3) & 1;
    const int nb = (b >> 4) * 8 + (b & 7);
    char* wsl = P.ws[rank];
    gu32* tmo = (gu32*)(wsl + WS_TMO);
    gu32* ctr = (gu32*)(wsl + WS_CTR + (size_t)b * 128);

    f32x4 acc[4][8];
    bool good = true;
    u32 call = 0;

    if (wave >= 4) {
        const int pw = wave - 4;
        const int h = lane >> 5, q = lane & 31;
        const u32x4 srdW = make_srd(P.w, W_BYTES);
        const u32x4 srdA = make_srd(wsl + WS_A, A_BYTES);
        u32 voffW[8];
#pragma unroll
        for (int j = 0; j < 8; ++j)
            voffW[j] = (u32)(16 * pw + 8 * h + j) * (NTOT * 4) + (u32)(nb * 128 + 4 * q) * 4;
        const u32 voffA = lane * 16 + pw * 8192 + mh * 32768;
        const u32 m0A = lds0 + LDS_A + pw * 8192;
        lchar* bw = smem + LDS_B + (pw >> 1) * 8192 + (q >> 4) * 4096 + (2 * (pw & 1) + h) * 256 + (q & 15) * 16;

        u32x4 r[BD][8];
#define LOADW(d, s_) do { int _s = (s_); u32 _so = _s < NST ? ((u32)kstage_of(_s, rank) << 22) : OOB; \
        _Pragma("unroll") for (int j = 0; j < 8; ++j) \
            asm volatile("buffer_load_dwordx4 %0, %1, %2, %3 offen" WNT \
                         : "+v"(r[d][j]) : "v"(voffW[j]), "s"(srdW), "s"(_so) : "memory"); } while (0)
#define LOADA(s_) do { int _s = (s_); u32 _so = _s < NST ? ((u32)kstage_of(_s, rank) << 16) : OOB; \
        u32 _m = m0A + (_s & 3) * 32768; \
        asm volatile("s_mov_b32 m0, %0\n\ts_nop 0\n\t" \
                     "buffer_load_dwordx4 %1, %2, %3 offen sc0 sc1 lds\n\t" \
                     "buffer_load_dwordx4 %1, %2, %3 offen offset:1024 sc0 sc1 lds\n\t" \
                     "buffer_load_dwordx4 %1, %2, %3 offen offset:2048 sc0 sc1 lds\n\t" \
                     "buffer_load_dwordx4 %1, %2, %3 offen offset:3072 sc0 sc1 lds" \
                     :: "s"(_m), "v"(voffA), "s"(srdA), "s"(_so) : "memory"); \
        asm volatile("s_mov_b32 m0, %0\n\ts_nop 0\n\t" \
                     "buffer_load_dwordx4 %1, %2, %3 offen sc0 sc1 lds\n\t" \
                     "buffer_load_dwordx4 %1, %2, %3 offen offset:1024 sc0 sc1 lds\n\t" \
                     "buffer_load_dwordx4 %1, %2, %3 offen offset:2048 sc0 sc1 lds\n\t" \
                     "buffer_load_dwordx4 %1, %2, %3 offen offset:3072 sc0 sc1 lds" \
                     :: "s"(_m + 4096), "v"(voffA), "s"(srdA), "s"(_so + 4096) : "memory"); } while (0)
#define WAITW(d, N) asm volatile("s_waitcnt vmcnt(" #N ")" \
        : "+v"(r[d][0]), "+v"(r[d][1]), "+v"(r[d][2]), "+v"(r[d][3]), \
          "+v"(r[d][4]), "+v"(r[d][5]), "+v"(r[d][6]), "+v"(r[d][7]) :: "memory")
#define WRITEB(d, s_) do { \
        _Pragma("unroll") for (int c = 0; c < 4; ++c) { \
            u32x4 o; \
            o[0] = pack_bf16(r[d][0][c], r[d][1][c]); o[1] = pack_bf16(r[d][2][c], r[d][3][c]); \
            o[2] = pack_bf16(r[d][4][c], r[d][5][c]); o[3] = pack_bf16(r[d][6][c], r[d][7][c]); \
            *(__attribute__((address_space(3))) u32x4*)(bw + ((s_) & 1) * 16384 + c * 1024) = o; \
        } } while (0)

#pragma unroll
        for (int d = 0; d < BD; ++d)
#pragma unroll
            for (int j = 0; j < 8; ++j) r[d][j] = u32x4{0, 0, 0, 0};
        LOADW(0, 0); LOADW(1, 1); LOADW(2, 2); LOADW(3, 3);
        BARRIER();
        BARRIER();
        LOADA(0); LOADA(1);
        asm volatile("s_waitcnt vmcnt(0)" ::: "memory");
        for (int s = 0; s < NST; s += BD) {
#define INTERVAL(d) do { \
            LOADA(s + d + 2); \
            WAITW(d, 56); \
            WRITEB(d, d); \
            LOADW(d, s + d + BD); \
            asm volatile("s_waitcnt vmcnt(40)" ::: "memory"); \
            BARRIER(); } while (0)
            INTERVAL(0); INTERVAL(1); INTERVAL(2); INTERVAL(3);
        }
        asm volatile("s_waitcnt vmcnt(0)" ::: "memory");
        BARRIER();
#pragma unroll
        for (int m = 0; m < 4; ++m)
#pragma unroll
            for (int n = 0; n < 8; ++n) acc[m][n] = f32x4{0, 0, 0, 0};
    } else {
        u32 cv = __hip_atomic_load(ctr, __ATOMIC_RELAXED, __HIP_MEMORY_SCOPE_AGENT);
        const int dst = (b + rank) & (NR - 1);
        const int jb = b >> 3;
        const bool local = (dst == rank);
        char* wsd = P.ws[dst];
        const float* xr = P.x + (size_t)(dst * MLOC + jb * 16 + (lane & 15)) * KC + (lane >> 4) * 8;
        char* adst = wsd + WS_A + (size_t)rank * 8 * 65536 + (jb >> 4) * 32768 + (jb & 15) * 2048 + lane * 16;
        u32x4 pk[5];
#define CVTFRAG(i, f) do { \
            u32x4 lo = *(const u32x4*)(xr + (f) * 32), hi = *(const u32x4*)(xr + (f) * 32 + 4); \
            pk[i][0] = pack_bf16(lo[0], lo[1]); pk[i][1] = pack_bf16(lo[2], lo[3]); \
            pk[i][2] = pack_bf16(hi[0], hi[1]); pk[i][3] = pack_bf16(hi[2], hi[3]); } while (0)
#define FRAGPTR(f) (adst + ((f) >> 1) * 65536 + ((f) & 1) * 1024)
        const int f1 = 4 * wave;
        CVTFRAG(0, wave);
        if (wave > 0) { CVTFRAG(1, f1); CVTFRAG(2, f1 + 1); CVTFRAG(3, f1 + 2); CVTFRAG(4, f1 + 3); }
        store16_sys(FRAGPTR(wave), pk[0]);
        if (local && wave > 0) {
            store16_sys(FRAGPTR(f1), pk[1]); store16_sys(FRAGPTR(f1 + 1), pk[2]);
            store16_sys(FRAGPTR(f1 + 2), pk[3]); store16_sys(FRAGPTR(f1 + 3), pk[4]);
        }
        call = __builtin_amdgcn_readfirstlane(cv) + 1;
        asm volatile("s_waitcnt vmcnt(0)" ::: "memory");
        BARRIER();
        gu32* fl0 = (gu32*)(wsd + WS_FLAG + (size_t)(0 * NR + rank) * 128) + jb;
        gu32* fl1 = (gu32*)(wsd + WS_FLAG + (size_t)(1 * NR + rank) * 128) + jb;
        if (wave == 1 && lane == 0) {
            __hip_atomic_store(fl0, call, __ATOMIC_RELAXED, __HIP_MEMORY_SCOPE_SYSTEM);
            if (local) __hip_atomic_store(fl1, call, __ATOMIC_RELAXED, __HIP_MEMORY_SCOPE_SYSTEM);
        }
        if (wave == 0) {
            const char* fp = wsl + WS_FLAG + (size_t)(((lane >> 3) & 1) * NR + rank) * 128 + (lane & 7) * 16;
            good = poll_flags(fp, lane < 16, call, tmo);
        }
        BARRIER();
        if (!local && wave > 0) {
            store16_sys(FRAGPTR(f1), pk[1]); store16_sys(FRAGPTR(f1 + 1), pk[2]);
            store16_sys(FRAGPTR(f1 + 2), pk[3]); store16_sys(FRAGPTR(f1 + 3), pk[4]);
        }

#pragma unroll
        for (int m = 0; m < 4; ++m)
#pragma unroll
            for (int n = 0; n < 8; ++n) acc[m][n] = f32x4{0, 0, 0, 0};

        typedef __attribute__((address_space(3))) const bf16x8 lfrag;
        lchar* abase = smem + LDS_A + wave * 8192 + lane * 16;
        lchar* bbase = smem + LDS_B + lane * 16;
        bf16x8 X0[4], W0[8], X1[4], W1[8];
#define READF(X, W, s_, kk) do { \
            lchar* _a = abase + ((s_) & 3) * 32768 + (kk) * 1024; \
            lchar* _b = bbase + ((s_) & 1) * 16384 + (kk) * 8192; \
            _Pragma("unroll") for (int m = 0; m < 4; ++m) X[m] = *(lfrag*)(_a + m * 2048); \
            _Pragma("unroll") for (int n = 0; n < 8; ++n) W[n] = *(lfrag*)(_b + n * 1024); } while (0)
#define MMA(X, W) do { \
            _Pragma("unroll") for (int n = 0; n < 8; ++n) \
            _Pragma("unroll") for (int m = 0; m < 4; ++m) \
                acc[m][n] = __builtin_amdgcn_mfma_f32_16x16x32_bf16(W[n], X[m], acc[m][n], 0, 0, 0); } while (0)
#define SCHEDPAT() do { \
            _Pragma("unroll") for (int i = 0; i < 12; ++i) { \
                __builtin_amdgcn_sched_group_barrier(0x8, 2, 0); \
                __builtin_amdgcn_sched_group_barrier(0x100, 1, 0); } \
            __builtin_amdgcn_sched_group_barrier(0x8, 8, 0); } while (0)

        BARRIER();
        READF(X0, W0, 0, 0);
        for (int s = 0; s < NST; ++s) {
            __builtin_amdgcn_sched_barrier(0);
            READF(X1, W1, s, 1);
            MMA(X0, W0);
            SCHEDPAT();
            __builtin_amdgcn_sched_barrier(0);
            if (s == 4 || s == 18) {
                if (wave == 0) {
                    int src = (rank + 1 + (lane >> 3)) & (NR - 1);
                    const char* fp = wsl + WS_FLAG + (size_t)((s == 4 ? 0 : 1) * NR + src) * 128 + (lane & 7) * 16;
                    good = poll_flags(fp, lane < 56, call, tmo) && good;
                }
            }
            if (s == S_FLAGB) asm volatile("s_waitcnt vmcnt(0)" ::: "memory");
            BARRIER();
            if (s == S_FLAGB && !local && wave == 1 && lane == 0)
                __hip_atomic_store(fl1, call, __ATOMIC_RELAXED, __HIP_MEMORY_SCOPE_SYSTEM);
            __builtin_amdgcn_sched_barrier(0);
            READF(X0, W0, s + 1, 0);
            MMA(X1, W1);
            SCHEDPAT();
        }
    }

    typedef __attribute__((address_space(3))) float lfloat;
    typedef __attribute__((address_space(3))) u32 lu32;
    lfloat* red = (lfloat*)(smem + LDS_B);
    lu32* redu = (lu32*)(smem + LDS_B + 128);
    BARRIER();
    if (wave < 4) {
        float m = 0.f;
#pragma unroll
        for (int mi = 0; mi < 4; ++mi)
#pragma unroll
            for (int n = 0; n < 8; ++n)
                m = fmaxf(fmaxf(m, fmaxf(acc[mi][n][0], acc[mi][n][1])), fmaxf(acc[mi][n][2], acc[mi][n][3]));
        m = wave_max(m);
        if (lane == 0) {
            red[wave] = m;
            if (wave == 0) { redu[0] = call; redu[1] = good ? 0u : 1u; }
        }
    }
    BARRIER();
    call = redu[0];
    float amax;
    bool bad = redu[1] != 0;
    if (wave == 7 && lane < NR) {
        float m = fmaxf(fmaxf(red[0], red[1]), fmaxf(red[2], red[3]));
        gu64* g = (gu64*)(P.ws[lane & (NR - 1)] + WS_GRAN) + rank * NWG + b;
        __hip_atomic_store(g, ((u64)call << 32) | __builtin_bit_cast(u32, m), __ATOMIC_RELAXED,
                           __HIP_MEMORY_SCOPE_SYSTEM);
    }
    {
        gu64* g = (gu64*)(wsl + WS_GRAN) + wave * NWG + lane;
        float m = 0.f;
        bool wbad = false;
        u64 t0 = __builtin_amdgcn_s_memrealtime();
        for (;;) {
            u64 v0 = __hip_atomic_load(g, __ATOMIC_RELAXED, __HIP_MEMORY_SCOPE_SYSTEM);
            u64 v1 = __hip_atomic_load(g + 64, __ATOMIC_RELAXED, __HIP_MEMORY_SCOPE_SYSTEM);
            u64 v2 = __hip_atomic_load(g + 128, __ATOMIC_RELAXED, __HIP_MEMORY_SCOPE_SYSTEM);
            u64 v3 = __hip_atomic_load(g + 192, __ATOMIC_RELAXED, __HIP_MEMORY_SCOPE_SYSTEM);
            u32 tv = __hip_atomic_load(tmo, __ATOMIC_RELAXED, __HIP_MEMORY_SCOPE_AGENT);
            bool ok = (u32)(v0 >> 32) == call && (u32)(v1 >> 32) == call &&
                      (u32)(v2 >> 32) == call && (u32)(v3 >> 32) == call;
            m = fmaxf(fmaxf(__builtin_bit_cast(float, (u32)v0), __builtin_bit_cast(float, (u32)v1)),
                      fmaxf(__builtin_bit_cast(float, (u32)v2), __builtin_bit_cast(float, (u32)v3)));
            if (__builtin_amdgcn_readfirstlane(tv) == call) { wbad = true; break; }
            if (__all(ok)) break;
            if (__builtin_amdgcn_s_memrealtime() - t0 > DEADLINE_TICKS) {
                __hip_atomic_store(tmo, call, __ATOMIC_RELAXED, __HIP_MEMORY_SCOPE_AGENT);
                wbad = true;
                break;
            }
            __builtin_amdgcn_s_sleep(2);
        }
        m = wave_max(m);
        if (lane == 0) { red[8 + wave] = m; redu[8 + wave] = wbad ? 1u : 0u; }
    }
    BARRIER();
    amax = 0.f;
#pragma unroll
    for (int i = 0; i < 8; ++i) { amax = fmaxf(amax, red[8 + i]); bad = bad || (redu[8 + i] != 0); }

    if (wave < 4) {
        const float scale = amax / 448.0f;
        const float inv = scale > 0.f ? 1.0f / scale : 0.f;
        unsigned char* o = P.out + (size_t)(mh * 256 + wave * 64 + (lane & 15)) * NTOT + nb * 128 + (lane >> 4) * 16;
#pragma unroll
        for (int mi = 0; mi < 4; ++mi)
#pragma unroll
            for (int qh = 0; qh < 2; ++qh) {
                u32x4 v;
#pragma unroll
                for (int t = 0; t < 4; ++t) {
                    float y0 = __builtin_amdgcn_fmed3f(acc[mi][qh * 4 + 0][t] * inv, 0.f, 448.f);
                    float y1 = __builtin_amdgcn_fmed3f(acc[mi][qh * 4 + 1][t] * inv, 0.f, 448.f);
                    float y2 = __builtin_amdgcn_fmed3f(acc[mi][qh * 4 + 2][t] * inv, 0.f, 448.f);
                    float y3 = __builtin_amdgcn_fmed3f(acc[mi][qh * 4 + 3][t] * inv, 0.f, 448.f);
                    int p = __builtin_amdgcn_cvt_pk_fp8_f32(y0, y1, 0, false);
                    p = __builtin_amdgcn_cvt_pk_fp8_f32(y2, y3, p, true);
                    v[t] = bad ? 0x7f7f7f7fu : (u32)p;
                }
                *(u32x4*)(o + (size_t)mi * 16 * NTOT + qh * 64) = v;
            }
    } else if (wave == 4 && lane == 0) {
        __hip_atomic_store(ctr, call, __ATOMIC_RELAXED, __HIP_MEMORY_SCOPE_AGENT);
    }
}

extern "C" void kernel_launch(int rank, int nranks, ncclComm_t comm,
                              hipStream_t stream,
                              void* const* d_in, const int* in_sizes, int n_in,
                              void* d_out, int out_size,
                              const int* in_dtypes, int out_dtype,
                              void* const* d_out_all,
                              void* const* d_ws_all, size_t ws_bytes) {
    if (nranks != NR || (unsigned)rank >= (unsigned)NR || n_in < 2 || ws_bytes < WS_END ||
        in_sizes[0] != KTOT * KC || in_sizes[1] != KTOT * NTOT || out_size != MLOC * NTOT) {
        fprintf(stderr, "kernel_launch: unexpected problem shape\n");
        return;
    }
    HIP_CHECK(hipFuncSetAttribute((const void*)a2a_gemm_kernel,
                                  hipFuncAttributeMaxDynamicSharedMemorySize, LDS_BYTES));
    Params p;
    p.x = (const float*)d_in[0];
    p.w = (const float*)d_in[1];
    p.out = (unsigned char*)d_out;
    for (int i = 0; i < NR; ++i) p.ws[i] = (char*)d_ws_all[i];
    p.rank = rank;
    p.pad = 0;
    hipLaunchKernelGGL(a2a_gemm_kernel, dim3(NWG), dim3(NTHR), LDS_BYTES, stream, p);
    HIP_CHECK(hipGetLastError());
}
```

```cpp
#define W_NT 1
#define PTAB 1
#define pT 150
#define pT0 580
#define pTE 56
#define pK0 12
#define pT1 200
#define pTS 7
#define pS 50
#define NG 16
#define PGRP (nb>>3)
#define TAIL 1
#define GATH2 1
#define VP -1
#define WAHEAD 6
#define EPK 1
#define CS 1
#define OUTP "sc0 sc1"
#include <hip/hip_runtime.h>
#include <rccl/rccl.h>
#include <cstdio>
#include <cstdint>

#define HIP_CHECK(expr) do { hipError_t _e = (expr); if (_e != hipSuccess) { \
    fprintf(stderr, "HIP error %s at %s:%d\n", hipGetErrorString(_e), __FILE__, __LINE__); \
    return; } } while (0)

typedef unsigned u32;
typedef unsigned long long u64;
typedef unsigned u32x4 __attribute__((ext_vector_type(4)));
typedef float f32x2 __attribute__((ext_vector_type(2)));
typedef float f32x4 __attribute__((ext_vector_type(4)));
typedef float f32x16 __attribute__((ext_vector_type(16)));
typedef __bf16 bf16x2 __attribute__((ext_vector_type(2)));
typedef __bf16 bf16x8 __attribute__((ext_vector_type(8)));
typedef __attribute__((address_space(1))) u32 gu32;
typedef __attribute__((address_space(1))) u64 gu64;
typedef __attribute__((address_space(3))) char lchar;
typedef __attribute__((address_space(3))) float lfloat;
typedef __attribute__((address_space(3))) u32 lu32;
typedef __attribute__((address_space(3))) u32x4 lu32x4;
typedef __attribute__((address_space(3))) const bf16x8 lfrag;

constexpr int NR = 8;
constexpr int MLOC = 512;
constexpr int KTOT = 4096;
constexpr int NTOT = 16384;
constexpr int KC = 512;
constexpr int NWG = 256;
constexpr int NTHR = 512;
constexpr int NST = 64;

constexpr u32 A_BYTES = (u32)MLOC * KTOT * 2;
constexpr size_t WS_FLAG = A_BYTES;
constexpr u32 AF_BYTES = A_BYTES + 8192 + 256;
constexpr size_t WS_GRAN = WS_FLAG + 65536;
constexpr size_t WS_CTR = WS_GRAN + 65536;
constexpr size_t WS_TMO = WS_CTR + 65536;
constexpr size_t WS_XF = WS_TMO + 4096;
constexpr size_t WS_STAMP = WS_XF + 262144;
constexpr int NSTAMP = 24;
constexpr size_t WS_X = (size_t)8 << 20;
constexpr size_t WS_X2 = WS_X + ((size_t)16 << 20);
constexpr size_t WS_END = WS_X2 + ((size_t)8 << 20);
#define STAMPW(i, w) do {} while (0)
#define STAMPC(i, t0) do {} while (0)
#define STAMP(i) STAMPW(i, 0)
#define STAMP_E
#define STAMP_E1
#define STAMP_E2
#define EST(i)
#define STAMP_P(i)
#define STAMP_C0
#define STAMP_G
#define STAMP_V
#define STAMP_S
#define STAMP_U(i)
#define LST_0
#define LST_A
#define LST_B
constexpr u32 W_BYTES = (u32)KTOT * NTOT * 4;

constexpr u32 LDS_RED = 2560;
constexpr u32 LDS_RING = 32768;
constexpr u32 LDS_BYTES = LDS_RING + 65536 + 32768;

constexpr u64 DEADLINE_TICKS = 50000000ull;
constexpr u32 OOB = 0x80000000u;
constexpr int SPIN_CAP = 1 << 24;

#define WNT " nt"

#define PCONV 0
#define DLOOP "eloop.inc"
typedef u32 u32x16 __attribute__((ext_vector_type(16)));
#define XSTR2(x) #x
#define XSTR(x) XSTR2(x)
#define CSX (ro ^ 1)
struct Params {
    const float* x;
    const float* w;
    unsigned char* out;
    char* ws[NR];
    int rank;
    int pad;
};

__device__ __forceinline__ u32 pack_bf16(u32 a, u32 b) {
    f32x2 f = {__builtin_bit_cast(float, a), __builtin_bit_cast(float, b)};
    return __builtin_bit_cast(u32, __builtin_convertvector(f, bf16x2));
}
__device__ __forceinline__ u32 pack_bf16f(float a, float b) {
    f32x2 f = {a, b};
    return __builtin_bit_cast(u32, __builtin_convertvector(f, bf16x2));
}

__device__ __forceinline__ u32x4 make_srd(const void* p, u32 bytes) {
    u64 a = (u64)p;
    u32x4 r;
    r[0] = __builtin_amdgcn_readfirstlane((u32)a);
    r[1] = __builtin_amdgcn_readfirstlane((u32)(a >> 32)) & 0xffffu;
    r[2] = bytes;
    r[3] = 0x00020000u;
    return r;
}

#define BARRIER() asm volatile("s_waitcnt lgkmcnt(0)\n\ts_barrier" ::: "memory")
#define CBAR() asm volatile("" ::: "memory")

__device__ __forceinline__ void store16_sys(char* p, u32x4 v) {
    asm volatile("global_store_dwordx4 %0, %1, off sc0 sc1\n\ts_nop 1" :: "v"(p), "v"(v) : "memory");
}

__device__ __forceinline__ float wave_max(float m) {
    m = fmaxf(m, __shfl_xor(m, 32));
    m = fmaxf(m, __shfl_xor(m, 16));
    m = fmaxf(m, __shfl_xor(m, 8));
    m = fmaxf(m, __shfl_xor(m, 4));
    m = fmaxf(m, __shfl_xor(m, 2));
    m = fmaxf(m, __shfl_xor(m, 1));
    return m;
}

__device__ __forceinline__ u32 wave_maxu(u32 m) {
    m = max(m, (u32)__builtin_amdgcn_update_dpp(0, (int)m, 0x111, 0xf, 0xf, true));
    m = max(m, (u32)__builtin_amdgcn_update_dpp(0, (int)m, 0x112, 0xf, 0xf, true));
    m = max(m, (u32)__builtin_amdgcn_update_dpp(0, (int)m, 0x114, 0xf, 0xf, true));
    m = max(m, (u32)__builtin_amdgcn_update_dpp(0, (int)m, 0x118, 0xf, 0xf, true));
    return max(max((u32)__builtin_amdgcn_readlane((int)m, 15), (u32)__builtin_amdgcn_readlane((int)m, 31)),
               max((u32)__builtin_amdgcn_readlane((int)m, 47), (u32)__builtin_amdgcn_readlane((int)m, 63)));
}

__global__ __launch_bounds__(NTHR) void a2a_gemm_kernel(Params P) {
    extern __shared__ __attribute__((aligned(16))) char smem_g[];
    lchar* smem = (lchar*)smem_g;

    const int b = blockIdx.x;
    u32 cp0, cp1;
    asm volatile("s_getpc_b64 s[40:41]\na2a_code_begin:\n\tv_lshlrev_b32 %0, 7, %2\n\tv_add_u32 %1, 0x2000, %0\n\tv_min_u32 %1, a2a_code_end-a2a_code_begin, %1\n\t"
                 "global_load_dword %0, %0, s[40:41]\n\tglobal_load_dword %1, %1, s[40:41]"
                 : "=&{v243}"(cp0), "=&{v242}"(cp1) : "v"(threadIdx.x & 63) : "s40", "s41");
    const int rank = P.rank & (NR - 1);
    const int wave = __builtin_amdgcn_readfirstlane(threadIdx.x >> 6);
    const int ro = (b >> 2) & 1;
#define SHAPE 0
    const int nb = ((b >> 3) & 0x18) * 4 + (b & 1) * 16 + ((b >> 3) & 7) * 2 + ((b >> 1) & 1);
    const int col0 = nb * 128, cst = 64;
    char* wsl = P.ws[0];
    gu32* tmo = (gu32*)(wsl + WS_TMO);
    gu32* ctr = (gu32*)(wsl + WS_CTR + (size_t)b * 128);
    lu32* redu = (lu32*)(smem + LDS_RED);

    u32 pl;
    u32 call, l4, eu0, eu1, eu2;
    char* wsw = P.ws[wave];
    unsigned char* outp = P.out;
    f32x16 av[8];
    {
        const int lane = threadIdx.x & 63;
        STAMP(0);
        const int ch = (b >> 3) & 7, qr = b >> 6;
        const int dd = (b + ch + 2 * qr) & 7;
        const int dst = (rank + dd) & (NR - 1);
        const int jb = qr * 8 + wave;
        char* wsd = P.ws[dd];
        const int sd = dd == 0 ? ch : 15 + 7 * ch - dd;
        gu32* fl = (gu32*)(wsd + WS_FLAG + sd * 128 + qr * 4);
        char* pu = wsd + (size_t)sd * 65536 + jb * 1024 - wave * 4096;
#define PUSHM 2
#define URG 1
        const int pp = (ch - URG) * PUSHM;
        const int tp = (dd == 0 || ch < URG) ? 0 : 2 * pp + 2;
        const int tf = (tp > 1 && wave == (pp & 1) * 4) ? tp + 4 : 1;
        const int td = tp > 1 ? max(tp - (wave < 4 ? 6 : 4), 0) : 1;
        const u64 ev = wave == 0 ? ~0ull : ((1ull << tp) | (1ull << tf) | (1ull << td) | (1ull << (2 * VP + 2))
            );
        const float* xp = P.x + (size_t)(dst * MLOC + jb * 16) * KC + ch * 64;
        const u32x4 srdA = make_srd(wsl, AF_BYTES);
        const u32x4 srdW = make_srd(P.w, W_BYTES);
        const int ww = wave & 1, h = lane >> 5, q = lane & 31;
        const u32 voffW = (u32)(16 * ww + 8 * h) * (NTOT * 4) + (u32)(col0 + ((q >> 4) ^ CSX) * cst + 4 * (q & 15)) * 4;
        u32 wa = LDS_RING + (wave >> 1) * 8192 + (q >> 4) * 4096 + (2 * ww + h) * 256 + (q & 15) * 16;
        u32 ab = (lane & 15) * 2048 + (lane >> 4) * 32;
        u32 ktab;
        {
            int t = lane - 8, c = t / 7, d = t - c * 7;
            ktab = (u32)(lane < 8 ? rank * 8 + lane : ((rank + 1 + d) & 7) * 8 + c) << 22;
        }
        const u64 kh0 = 0xccccccccccccccccull ^ (((b ^ ro) & 1) ? ~0ull : 0ull);
        const u32 kf = (b >> 3) & 1;
        const u32 po = ((u32)(((rank + 4) & 7) * 8 + 4) << 22) + (kf << 21) + ro * (8 << 16) + nb * 512;
        u32 pd, ls;
        const u32 vp = lane < 4 ? (lane >> 1) * 65536 + (((b >> 3) * 4 + (wave & 3)) * 2 + (lane & 1)) * 128 : OOB;
        l4 = lane * 4;
        const int pk = lane >> 1, pk1 = pTS / 2;
        const int pr = min(max(pk - pK0, 0), (pT1 - pT) / pTS);
        const int pst = (2 * (PGRP) - (NG - 1)) * pS * (pT1 - pTS * pr - pT) / (2 * (pT1 - pT)) * 4 / NG;
        const u32 dl = lane > pTE ? 0 : pst + pT0 + pT1 * (min(pk, pK0) + pr) - pTS * pr * (pr + 1) / 2 + pT * (max(pk - pK0, 0) - pr);
        asm volatile(
        "v_mov_b32 v180, 0\n\t"
        "global_load_dword %[pl], v180, %[ctr] sc1\n\t"
        "s_cmp_lg_u32 %[tp], 0\n\t"
        "s_cbranch_scc1 16f\n\t"
        "v_lshrrev_b32 v156, 6, %[l4]\n\t"
        "v_and_b32 v157, 60, %[l4]\n\t"
        "v_lshlrev_b32 v152, 11, v156\n\t"
        "v_lshl_add_u32 v152, v157, 2, v152\n\t"
        "global_load_dwordx4 v[128:131], v152, %[xp]\n\t"
        "v_add_u32 v153, 8192, v152\n\t"
        "global_load_dwordx4 v[132:135], v153, %[xp]\n\t"
        "v_add_u32 v154, 16384, v152\n\t"
        "global_load_dwordx4 v[136:139], v154, %[xp]\n\t"
        "v_add_u32 v155, 24576, v152\n\t"
        "global_load_dwordx4 v[140:143], v155, %[xp]\n\t"
        "s_lshl_b32 s49, %[wv], 12\n\t"
        "v_lshl_add_u32 v156, v156, 4, s49\n\t"
        "v_and_b32 v157, 56, %[l4]\n\t"
        "v_lshl_add_u32 v156, v157, 5, v156\n\t"
        "v_and_b32 v157, 4, %[l4]\n\t"
        "v_lshl_add_u32 v156, v157, 1, v156\n\t"
        "16:\n\t"
        "s_cmp_lg_u32 %[td], 0\n\t"
        "s_cbranch_scc1 21f\n\t"
        "s_lshl_b32 s49, %[wv], 12\n\t"
        "s_add_u32 m0, s49, 98304\n\t"
        "s_nop 0\n\t"
        "global_load_lds_dwordx4 %[ab], %[xp] offset:0\n\t"
        "s_add_u32 m0, s49, 99312\n\t"
        "s_nop 0\n\t"
        "global_load_lds_dwordx4 %[ab], %[xp] offset:16\n\t"
        "s_add_u32 m0, s49, 100224\n\t"
        "s_nop 0\n\t"
        "global_load_lds_dwordx4 %[ab], %[xp] offset:128\n\t"
        "s_add_u32 m0, s49, 101232\n\t"
        "s_nop 0\n\t"
        "global_load_lds_dwordx4 %[ab], %[xp] offset:144\n\t"
        "21:\n\t"
        "s_mov_b64 s[44:45], %[kh0]\n\t"
        "s_memrealtime s[54:55]\n\t"
        "s_waitcnt lgkmcnt(0)\n\t"
        "s_mov_b32 s59, s54\n\t"
        "s_mov_b32 s53, 0\n\t"
        "s_mov_b32 s42, 0\n\t"
        "s_lshr_b32 s50, %[wv], 1\n\t"
        "v_readlane_b32 s48, %[kt], s50\n\t"
        "s_bitcmp1_b64 s[44:45], s50\n\t"
        "s_cselect_b32 s49, 0x200000, 0\n\t"
        "s_add_u32 s48, s48, s49\n\t"
        "s_cmp_lt_u32 s50, 64\n\t"
        "s_cselect_b32 s48, s48, 0x80000000\n\t"
        "s_cmp_lg_u32 %[wv], 0\n\t"
        "s_cbranch_scc1 1f\n\t"
        "ds_write_b32 %[l4], v180 offset:2560\n\t"
        "1:\n\t"
        "s_cmp_eq_u32 %[tp], 0\n\t"
        "s_cbranch_scc1 2f\n\t"
        "s_cmp_eq_u32 %[wv], 3\n\t"
        "s_cbranch_scc1 5f\n\t"
        "s_cmp_lt_u32 %[wv], 4\n\t"
        "s_cselect_b64 vcc, 0, -1\n\t"
        "buffer_load_dwordx4 v[208:211], %[vw], %[srdw], s48 offen" WNT "\n\t"
        "s_add_u32 s48, s48, 0x10000\n\t"
        "v_mov_b64 v[0:1], 0\n\t"
        "v_mov_b64 v[2:3], 0\n\t"
        "v_mov_b64 v[4:5], 0\n\t"
        "v_mov_b64 v[6:7], 0\n\t"
        "v_mov_b64 v[8:9], 0\n\t"
        "v_mov_b64 v[10:11], 0\n\t"
        "v_mov_b64 v[12:13], 0\n\t"
        "v_mov_b64 v[14:15], 0\n\t"
        "s_sleep 5\n\t"
        "s_cbranch_vccnz 20f\n\t"
        "s_sleep 5\n\t"
        "20:\n\t"
        "buffer_load_dwordx4 v[212:215], %[vw], %[srdw], s48 offen" WNT "\n\t"
        "s_add_u32 s48, s48, 0x10000\n\t"
        "v_mov_b64 v[16:17], 0\n\t"
        "v_mov_b64 v[18:19], 0\n\t"
        "v_mov_b64 v[20:21], 0\n\t"
        "v_mov_b64 v[22:23], 0\n\t"
        "v_mov_b64 v[24:25], 0\n\t"
        "v_mov_b64 v[26:27], 0\n\t"
        "v_mov_b64 v[28:29], 0\n\t"
        "v_mov_b64 v[30:31], 0\n\t"
        "s_sleep 5\n\t"
        "s_cbranch_vccnz 20f\n\t"
        "s_sleep 5\n\t"
        "20:\n\t"
        "buffer_load_dwordx4 v[216:219], %[vw], %[srdw], s48 offen" WNT "\n\t"
        "s_add_u32 s48, s48, 0x10000\n\t"
        "v_mov_b64 v[32:33], 0\n\t"
        "v_mov_b64 v[34:35], 0\n\t"
        "v_mov_b64 v[36:37], 0\n\t"
        "v_mov_b64 v[38:39], 0\n\t"
        "v_mov_b64 v[40:41], 0\n\t"
        "v_mov_b64 v[42:43], 0\n\t"
        "v_mov_b64 v[44:45], 0\n\t"
        "v_mov_b64 v[46:47], 0\n\t"
        "s_sleep 5\n\t"
        "s_cbranch_vccnz 20f\n\t"
        "s_sleep 5\n\t"
        "20:\n\t"
        "buffer_load_dwordx4 v[220:223], %[vw], %[srdw], s48 offen" WNT "\n\t"
        "s_add_u32 s48, s48, 0x10000\n\t"
        "v_mov_b64 v[48:49], 0\n\t"
        "v_mov_b64 v[50:51], 0\n\t"
        "v_mov_b64 v[52:53], 0\n\t"
        "v_mov_b64 v[54:55], 0\n\t"
        "v_mov_b64 v[56:57], 0\n\t"
        "v_mov_b64 v[58:59], 0\n\t"
        "v_mov_b64 v[60:61], 0\n\t"
        "v_mov_b64 v[62:63], 0\n\t"
        "s_sleep 5\n\t"
        "s_cbranch_vccnz 20f\n\t"
        "s_sleep 5\n\t"
        "20:\n\t"
        "buffer_load_dwordx4 v[224:227], %[vw], %[srdw], s48 offen" WNT "\n\t"
        "s_add_u32 s48, s48, 0x10000\n\t"
        "v_mov_b64 v[64:65], 0\n\t"
        "v_mov_b64 v[66:67], 0\n\t"
        "v_mov_b64 v[68:69], 0\n\t"
        "v_mov_b64 v[70:71], 0\n\t"
        "v_mov_b64 v[72:73], 0\n\t"
        "v_mov_b64 v[74:75], 0\n\t"
        "v_mov_b64 v[76:77], 0\n\t"
        "v_mov_b64 v[78:79], 0\n\t"
        "s_sleep 5\n\t"
        "s_cbranch_vccnz 20f\n\t"
        "s_sleep 5\n\t"
        "20:\n\t"
        "buffer_load_dwordx4 v[228:231], %[vw], %[srdw], s48 offen" WNT "\n\t"
        "s_add_u32 s48, s48, 0x10000\n\t"
        "v_mov_b64 v[80:81], 0\n\t"
        "v_mov_b64 v[82:83], 0\n\t"
        "v_mov_b64 v[84:85], 0\n\t"
        "v_mov_b64 v[86:87], 0\n\t"
        "v_mov_b64 v[88:89], 0\n\t"
        "v_mov_b64 v[90:91], 0\n\t"
        "v_mov_b64 v[92:93], 0\n\t"
        "v_mov_b64 v[94:95], 0\n\t"
        "s_sleep 5\n\t"
        "s_cbranch_vccnz 20f\n\t"
        "s_sleep 5\n\t"
        "20:\n\t"
        "buffer_load_dwordx4 v[232:235], %[vw], %[srdw], s48 offen" WNT "\n\t"
        "s_add_u32 s48, s48, 0x10000\n\t"
        "v_mov_b64 v[96:97], 0\n\t"
        "v_mov_b64 v[98:99], 0\n\t"
        "v_mov_b64 v[100:101], 0\n\t"
        "v_mov_b64 v[102:103], 0\n\t"
        "v_mov_b64 v[104:105], 0\n\t"
        "v_mov_b64 v[106:107], 0\n\t"
        "v_mov_b64 v[108:109], 0\n\t"
        "v_mov_b64 v[110:111], 0\n\t"
        "s_sleep 5\n\t"
        "s_cbranch_vccnz 20f\n\t"
        "s_sleep 5\n\t"
        "20:\n\t"
        "buffer_load_dwordx4 v[236:239], %[vw], %[srdw], s48 offen" WNT "\n\t"
        "v_mov_b64 v[112:113], 0\n\t"
        "v_mov_b64 v[114:115], 0\n\t"
        "v_mov_b64 v[116:117], 0\n\t"
        "v_mov_b64 v[118:119], 0\n\t"
        "v_mov_b64 v[120:121], 0\n\t"
        "v_mov_b64 v[122:123], 0\n\t"
        "v_mov_b64 v[124:125], 0\n\t"
        "v_mov_b64 v[126:127], 0\n\t"
        "s_waitcnt vmcnt(8)\n\t"
        "v_readfirstlane_b32 s43, %[pl]\n\t"
        "s_add_u32 s43, s43, 1\n\t"
        "s_branch 3f\n\t"
        "5:\n\t"
        "v_lshlrev_b32 v176, 14, %[l4]\n\t"
        "s_xor_b32 s50, %[po], 0x200000\n\t"
        "s_mov_b64 exec, 0xff\n\t"
        "buffer_load_dword v177, v176, %[srdw], %[po] offen\n\t"
        "buffer_load_dword v178, v176, %[srdw], s50 offen\n\t"
        "s_mov_b64 exec, -1\n\t"
        "v_mov_b64 v[0:1], 0\n\t"
        "v_mov_b64 v[2:3], 0\n\t"
        "v_mov_b64 v[4:5], 0\n\t"
        "v_mov_b64 v[6:7], 0\n\t"
        "v_mov_b64 v[8:9], 0\n\t"
        "v_mov_b64 v[10:11], 0\n\t"
        "v_mov_b64 v[12:13], 0\n\t"
        "v_mov_b64 v[14:15], 0\n\t"
        "v_mov_b64 v[16:17], 0\n\t"
        "v_mov_b64 v[18:19], 0\n\t"
        "v_mov_b64 v[20:21], 0\n\t"
        "v_mov_b64 v[22:23], 0\n\t"
        "v_mov_b64 v[24:25], 0\n\t"
        "v_mov_b64 v[26:27], 0\n\t"
        "v_mov_b64 v[28:29], 0\n\t"
        "v_mov_b64 v[30:31], 0\n\t"
        "v_mov_b64 v[32:33], 0\n\t"
        "v_mov_b64 v[34:35], 0\n\t"
        "v_mov_b64 v[36:37], 0\n\t"
        "v_mov_b64 v[38:39], 0\n\t"
        "v_mov_b64 v[40:41], 0\n\t"
        "v_mov_b64 v[42:43], 0\n\t"
        "v_mov_b64 v[44:45], 0\n\t"
        "v_mov_b64 v[46:47], 0\n\t"
        "v_mov_b64 v[48:49], 0\n\t"
        "v_mov_b64 v[50:51], 0\n\t"
        "v_mov_b64 v[52:53], 0\n\t"
        "v_mov_b64 v[54:55], 0\n\t"
        "v_mov_b64 v[56:57], 0\n\t"
        "v_mov_b64 v[58:59], 0\n\t"
        "v_mov_b64 v[60:61], 0\n\t"
        "v_mov_b64 v[62:63], 0\n\t"
        "v_mov_b64 v[64:65], 0\n\t"
        "v_mov_b64 v[66:67], 0\n\t"
        "v_mov_b64 v[68:69], 0\n\t"
        "v_mov_b64 v[70:71], 0\n\t"
        "v_mov_b64 v[72:73], 0\n\t"
        "v_mov_b64 v[74:75], 0\n\t"
        "v_mov_b64 v[76:77], 0\n\t"
        "v_mov_b64 v[78:79], 0\n\t"
        "v_mov_b64 v[80:81], 0\n\t"
        "v_mov_b64 v[82:83], 0\n\t"
        "v_mov_b64 v[84:85], 0\n\t"
        "v_mov_b64 v[86:87], 0\n\t"
        "v_mov_b64 v[88:89], 0\n\t"
        "v_mov_b64 v[90:91], 0\n\t"
        "v_mov_b64 v[92:93], 0\n\t"
        "v_mov_b64 v[94:95], 0\n\t"
        "v_mov_b64 v[96:97], 0\n\t"
        "v_mov_b64 v[98:99], 0\n\t"
        "v_mov_b64 v[100:101], 0\n\t"
        "v_mov_b64 v[102:103], 0\n\t"
        "v_mov_b64 v[104:105], 0\n\t"
        "v_mov_b64 v[106:107], 0\n\t"
        "v_mov_b64 v[108:109], 0\n\t"
        "v_mov_b64 v[110:111], 0\n\t"
        "v_mov_b64 v[112:113], 0\n\t"
        "v_mov_b64 v[114:115], 0\n\t"
        "v_mov_b64 v[116:117], 0\n\t"
        "v_mov_b64 v[118:119], 0\n\t"
        "v_mov_b64 v[120:121], 0\n\t"
        "v_mov_b64 v[122:123], 0\n\t"
        "v_mov_b64 v[124:125], 0\n\t"
        "v_mov_b64 v[126:127], 0\n\t"
        "s_waitcnt vmcnt(0)\n\t"
        "s_mov_b64 exec, 0xff\n\t"
        "buffer_load_dword v177, v176, %[srdw], %[po] offen offset:256\n\t"
        "buffer_load_dword v178, v176, %[srdw], s50 offen offset:256\n\t"
        "s_mov_b64 exec, -1\n\t"
        "buffer_load_dwordx4 v[208:211], %[vw], %[srdw], s48 offen" WNT "\n\t"
        "s_add_u32 s48, s48, 0x10000\n\t"
        "buffer_load_dwordx4 v[212:215], %[vw], %[srdw], s48 offen" WNT "\n\t"
        "s_add_u32 s48, s48, 0x10000\n\t"
        "buffer_load_dwordx4 v[216:219], %[vw], %[srdw], s48 offen" WNT "\n\t"
        "s_add_u32 s48, s48, 0x10000\n\t"
        "buffer_load_dwordx4 v[220:223], %[vw], %[srdw], s48 offen" WNT "\n\t"
        "s_add_u32 s48, s48, 0x10000\n\t"
        "buffer_load_dwordx4 v[224:227], %[vw], %[srdw], s48 offen" WNT "\n\t"
        "s_add_u32 s48, s48, 0x10000\n\t"
        "buffer_load_dwordx4 v[228:231], %[vw], %[srdw], s48 offen" WNT "\n\t"
        "s_add_u32 s48, s48, 0x10000\n\t"
        "buffer_load_dwordx4 v[232:235], %[vw], %[srdw], s48 offen" WNT "\n\t"
        "s_add_u32 s48, s48, 0x10000\n\t"
        "buffer_load_dwordx4 v[236:239], %[vw], %[srdw], s48 offen" WNT "\n\t"
        "v_readfirstlane_b32 s43, %[pl]\n\t"
        "s_add_u32 s43, s43, 1\n\t"
        "s_waitcnt vmcnt(9)\n\t"
        "s_memtime s[56:57]\n\t"
        "s_waitcnt vmcnt(8)\n\t"
        "s_memtime s[54:55]\n\t"
        "s_waitcnt lgkmcnt(0)\n\t"
        "s_sub_u32 s49, s54, s56\n\t"
        "s_cmp_gt_u32 s49, 110\n\t"
        "s_cselect_b32 s49, 0, 1\n\t"
        "s_xor_b32 s49, s49, %[vb]\n\t"
        "s_getreg_b32 s50, hwreg(HW_REG_XCC_ID, 2, 1)\n\t"
        "s_xor_b32 s49, s49, s50\n\t"
        "s_lshl1_add_u32 s49, s50, s49\n\t"
        "s_lshl2_add_u32 s49, s43, s49\n\t"
        "v_mov_b32 v180, %[vo]\n\t"
        "v_mov_b32 v181, s49\n\t"
        "s_mov_b64 exec, 1\n\t"
        "buffer_store_byte v181, v180, %[srda], 0 offen sc1\n\t"
        "s_mov_b64 exec, -1\n\t"
        "s_branch 3f\n\t"
        "2:\n\t"
        "v_mov_b64 v[0:1], 0\n\t"
        "v_mov_b64 v[2:3], 0\n\t"
        "v_mov_b64 v[4:5], 0\n\t"
        "v_mov_b64 v[6:7], 0\n\t"
        "v_mov_b64 v[8:9], 0\n\t"
        "v_mov_b64 v[10:11], 0\n\t"
        "v_mov_b64 v[12:13], 0\n\t"
        "v_mov_b64 v[14:15], 0\n\t"
        "v_mov_b64 v[16:17], 0\n\t"
        "v_mov_b64 v[18:19], 0\n\t"
        "v_mov_b64 v[20:21], 0\n\t"
        "v_mov_b64 v[22:23], 0\n\t"
        "v_mov_b64 v[24:25], 0\n\t"
        "v_mov_b64 v[26:27], 0\n\t"
        "v_mov_b64 v[28:29], 0\n\t"
        "v_mov_b64 v[30:31], 0\n\t"
        "v_mov_b64 v[32:33], 0\n\t"
        "v_mov_b64 v[34:35], 0\n\t"
        "v_mov_b64 v[36:37], 0\n\t"
        "v_mov_b64 v[38:39], 0\n\t"
        "v_mov_b64 v[40:41], 0\n\t"
        "v_mov_b64 v[42:43], 0\n\t"
        "v_mov_b64 v[44:45], 0\n\t"
        "v_mov_b64 v[46:47], 0\n\t"
        "v_mov_b64 v[48:49], 0\n\t"
        "v_mov_b64 v[50:51], 0\n\t"
        "v_mov_b64 v[52:53], 0\n\t"
        "v_mov_b64 v[54:55], 0\n\t"
        "v_mov_b64 v[56:57], 0\n\t"
        "v_mov_b64 v[58:59], 0\n\t"
        "v_mov_b64 v[60:61], 0\n\t"
        "v_mov_b64 v[62:63], 0\n\t"
        "v_mov_b64 v[64:65], 0\n\t"
        "v_mov_b64 v[66:67], 0\n\t"
        "v_mov_b64 v[68:69], 0\n\t"
        "v_mov_b64 v[70:71], 0\n\t"
        "v_mov_b64 v[72:73], 0\n\t"
        "v_mov_b64 v[74:75], 0\n\t"
        "v_mov_b64 v[76:77], 0\n\t"
        "v_mov_b64 v[78:79], 0\n\t"
        "v_mov_b64 v[80:81], 0\n\t"
        "v_mov_b64 v[82:83], 0\n\t"
        "v_mov_b64 v[84:85], 0\n\t"
        "v_mov_b64 v[86:87], 0\n\t"
        "v_mov_b64 v[88:89], 0\n\t"
        "v_mov_b64 v[90:91], 0\n\t"
        "v_mov_b64 v[92:93], 0\n\t"
        "v_mov_b64 v[94:95], 0\n\t"
        "v_mov_b64 v[96:97], 0\n\t"
        "v_mov_b64 v[98:99], 0\n\t"
        "v_mov_b64 v[100:101], 0\n\t"
        "v_mov_b64 v[102:103], 0\n\t"
        "v_mov_b64 v[104:105], 0\n\t"
        "v_mov_b64 v[106:107], 0\n\t"
        "v_mov_b64 v[108:109], 0\n\t"
        "v_mov_b64 v[110:111], 0\n\t"
        "v_mov_b64 v[112:113], 0\n\t"
        "v_mov_b64 v[114:115], 0\n\t"
        "v_mov_b64 v[116:117], 0\n\t"
        "v_mov_b64 v[118:119], 0\n\t"
        "v_mov_b64 v[120:121], 0\n\t"
        "v_mov_b64 v[122:123], 0\n\t"
        "v_mov_b64 v[124:125], 0\n\t"
        "v_mov_b64 v[126:127], 0\n\t"
        "s_waitcnt vmcnt(0)\n\t"
        "v_cvt_pk_bf16_f32 v128, v128, v129\n\t"
        "v_cvt_pk_bf16_f32 v129, v130, v131\n\t"
        "ds_write_b64 v156, v[128:129] offset:0\n\t"
        "v_cvt_pk_bf16_f32 v132, v132, v133\n\t"
        "v_cvt_pk_bf16_f32 v133, v134, v135\n\t"
        "ds_write_b64 v156, v[132:133] offset:64\n\t"
        "v_cvt_pk_bf16_f32 v136, v136, v137\n\t"
        "v_cvt_pk_bf16_f32 v137, v138, v139\n\t"
        "ds_write_b64 v156, v[136:137] offset:128\n\t"
        "v_cvt_pk_bf16_f32 v140, v140, v141\n\t"
        "v_cvt_pk_bf16_f32 v141, v142, v143\n\t"
        "ds_write_b64 v156, v[140:141] offset:192\n\t"
        "ds_read_b128 v[144:147], %[vx]\n\t"
        "ds_read_b128 v[148:151], %[vx] offset:1024\n\t"
        "s_waitcnt lgkmcnt(0)\n\t"
        "v_readfirstlane_b32 s43, %[pl]\n\t"
        "s_add_u32 s43, s43, 1\n\t"
        "v_add_u32 v182, 0x8000, %[vx]\n\t"
        "global_store_dwordx4 %[vx], v[144:147], %[pu] sc0 sc1\n\t"
        "global_store_dwordx4 v182, v[148:151], %[pu] sc0 sc1\n\t"
        "s_cmp_lg_u32 %[wv], 3\n\t"
        "s_cbranch_scc1 22f\n\t"
        "s_lshr_b32 s51, %[ld], 9\n\t"
        "s_getreg_b32 s49, hwreg(HW_REG_XCC_ID, 2, 1)\n\t"
        "s_xor_b32 s49, s49, s51\n\t"
        "s_and_b32 s49, s49, 1\n\t"
        "s_getreg_b32 s50, hwreg(HW_REG_XCC_ID, 2, 1)\n\t"
        "s_xor_b32 s49, s49, s50\n\t"
        "s_lshl1_add_u32 s49, s50, s49\n\t"
        "s_lshl2_add_u32 s49, s43, s49\n\t"
        "v_mov_b32 v180, %[vo]\n\t"
        "v_mov_b32 v181, s49\n\t"
        "s_mov_b64 exec, 1\n\t"
        "buffer_store_byte v181, v180, %[srda], 0 offen sc1\n\t"
        "s_mov_b64 exec, -1\n\t"
        "22:\n\t"
        "s_barrier\n\t"
        "s_cmp_lt_u32 %[wv], 4\n\t"
        "s_cbranch_scc0 13f\n\t"
        "buffer_load_dwordx4 v[208:211], %[vw], %[srdw], s48 offen" WNT "\n\t"
        "s_add_u32 s48, s48, 0x10000\n\t"
        "buffer_load_dwordx4 v[212:215], %[vw], %[srdw], s48 offen" WNT "\n\t"
        "s_add_u32 s48, s48, 0x10000\n\t"
        "buffer_load_dwordx4 v[216:219], %[vw], %[srdw], s48 offen" WNT "\n\t"
        "s_add_u32 s48, s48, 0x10000\n\t"
        "buffer_load_dwordx4 v[220:223], %[vw], %[srdw], s48 offen" WNT "\n\t"
        "s_add_u32 s48, s48, 0x10000\n\t"
        "buffer_load_dwordx4 v[224:227], %[vw], %[srdw], s48 offen" WNT "\n\t"
        "s_add_u32 s48, s48, 0x10000\n\t"
        "buffer_load_dwordx4 v[228:231], %[vw], %[srdw], s48 offen" WNT "\n\t"
        "s_add_u32 s48, s48, 0x10000\n\t"
        "buffer_load_dwordx4 v[232:235], %[vw], %[srdw], s48 offen" WNT "\n\t"
        "s_add_u32 s48, s48, 0x10000\n\t"
        "buffer_load_dwordx4 v[236:239], %[vw], %[srdw], s48 offen" WNT "\n\t"
        "s_waitcnt vmcnt(8)\n\t"
        "s_barrier\n\t"
        "s_branch 3f\n\t"
        "13:\n\t"
        "s_waitcnt vmcnt(0)\n\t"
        "s_barrier\n\t"
        "s_cmp_eq_u32 %[wv], 7\n\t"
        "s_cbranch_scc1 14f\n\t"
        "s_sleep 1\n\t"
        "s_branch 15f\n\t"
        "14:\n\t"
        "v_mov_b32 v180, 0\n\t"
        "v_mov_b32 v181, s43\n\t"
        "s_mov_b64 exec, 1\n\t"
        "global_store_dword v180, v181, %[fl] sc0 sc1\n\t"
        "s_mov_b64 exec, -1\n\t"
        "15:\n\t"
        "buffer_load_dwordx4 v[208:211], %[vw], %[srdw], s48 offen" WNT "\n\t"
        "s_add_u32 s48, s48, 0x10000\n\t"
        "buffer_load_dwordx4 v[212:215], %[vw], %[srdw], s48 offen" WNT "\n\t"
        "s_add_u32 s48, s48, 0x10000\n\t"
        "buffer_load_dwordx4 v[216:219], %[vw], %[srdw], s48 offen" WNT "\n\t"
        "s_add_u32 s48, s48, 0x10000\n\t"
        "buffer_load_dwordx4 v[220:223], %[vw], %[srdw], s48 offen" WNT "\n\t"
        "s_add_u32 s48, s48, 0x10000\n\t"
        "buffer_load_dwordx4 v[224:227], %[vw], %[srdw], s48 offen" WNT "\n\t"
        "s_add_u32 s48, s48, 0x10000\n\t"
        "buffer_load_dwordx4 v[228:231], %[vw], %[srdw], s48 offen" WNT "\n\t"
        "s_add_u32 s48, s48, 0x10000\n\t"
        "buffer_load_dwordx4 v[232:235], %[vw], %[srdw], s48 offen" WNT "\n\t"
        "s_add_u32 s48, s48, 0x10000\n\t"
        "buffer_load_dwordx4 v[236:239], %[vw], %[srdw], s48 offen" WNT "\n\t"
        "3:\n\t"
        "s_waitcnt vmcnt(0)\n\t"
        "v_cvt_pk_bf16_f32 v240, v208, v212\n\t"
        "v_cvt_pk_bf16_f32 v241, v216, v220\n\t"
        "v_cvt_pk_bf16_f32 v242, v224, v228\n\t"
        "v_cvt_pk_bf16_f32 v243, v232, v236\n\t"
        "ds_write_b128 %[wa], v[240:243] offset:0\n\t"
        "v_cvt_pk_bf16_f32 v240, v209, v213\n\t"
        "v_cvt_pk_bf16_f32 v241, v217, v221\n\t"
        "v_cvt_pk_bf16_f32 v242, v225, v229\n\t"
        "v_cvt_pk_bf16_f32 v243, v233, v237\n\t"
        "ds_write_b128 %[wa], v[240:243] offset:1024\n\t"
        "v_cvt_pk_bf16_f32 v240, v210, v214\n\t"
        "v_cvt_pk_bf16_f32 v241, v218, v222\n\t"
        "v_cvt_pk_bf16_f32 v242, v226, v230\n\t"
        "v_cvt_pk_bf16_f32 v243, v234, v238\n\t"
        "ds_write_b128 %[wa], v[240:243] offset:2048\n\t"
        "v_cvt_pk_bf16_f32 v240, v211, v215\n\t"
        "v_cvt_pk_bf16_f32 v241, v219, v223\n\t"
        "v_cvt_pk_bf16_f32 v242, v227, v231\n\t"
        "v_cvt_pk_bf16_f32 v243, v235, v239\n\t"
        "ds_write_b128 %[wa], v[240:243] offset:3072\n\t"
        "v_add_u32 %[wa], 0x8000, %[wa]\n\t"
        "s_mov_b32 s41, 64\n\t"
        "s_lshr_b32 s50, %[wv], 1\n\t"
        "s_add_u32 s50, s50, 4\n\t"
        "v_readlane_b32 s48, %[kt], s50\n\t"
        "s_bitcmp1_b64 s[44:45], s50\n\t"
        "s_cselect_b32 s49, 0x200000, 0\n\t"
        "s_add_u32 s48, s48, s49\n\t"
        "s_cmp_lt_u32 s50, 64\n\t"
        "s_cselect_b32 s48, s48, 0x80000000\n\t"
        "s_cmp_lt_u32 %[wv], 4\n\t"
        "s_cselect_b64 vcc, -1, 0\n\t"
        "buffer_load_dwordx4 v[208:211], %[vw], %[srdw], s48 offen" WNT "\n\t"
        "s_add_u32 s48, s48, 0x10000\n\t"
        "s_cbranch_vccnz 20f\n\t"
        "s_sleep 3\n\t"
        "20:\n\t"
        "buffer_load_dwordx4 v[212:215], %[vw], %[srdw], s48 offen" WNT "\n\t"
        "s_add_u32 s48, s48, 0x10000\n\t"
        "s_cbranch_vccnz 20f\n\t"
        "s_sleep 3\n\t"
        "20:\n\t"
        "buffer_load_dwordx4 v[216:219], %[vw], %[srdw], s48 offen" WNT "\n\t"
        "s_add_u32 s48, s48, 0x10000\n\t"
        "s_cbranch_vccnz 20f\n\t"
        "s_sleep 3\n\t"
        "20:\n\t"
        "buffer_load_dwordx4 v[220:223], %[vw], %[srdw], s48 offen" WNT "\n\t"
        "s_add_u32 s48, s48, 0x10000\n\t"
        "s_cbranch_vccnz 20f\n\t"
        "s_sleep 3\n\t"
        "20:\n\t"
        "buffer_load_dwordx4 v[224:227], %[vw], %[srdw], s48 offen" WNT "\n\t"
        "s_add_u32 s48, s48, 0x10000\n\t"
        "s_cbranch_vccnz 20f\n\t"
        "s_sleep 3\n\t"
        "20:\n\t"
        "buffer_load_dwordx4 v[228:231], %[vw], %[srdw], s48 offen" WNT "\n\t"
        "s_add_u32 s48, s48, 0x10000\n\t"
        "s_cbranch_vccnz 20f\n\t"
        "s_sleep 3\n\t"
        "20:\n\t"
        "buffer_load_dwordx4 v[232:235], %[vw], %[srdw], s48 offen" WNT "\n\t"
        "s_add_u32 s48, s48, 0x10000\n\t"
        "s_cbranch_vccnz 20f\n\t"
        "s_sleep 3\n\t"
        "20:\n\t"
        "buffer_load_dwordx4 v[236:239], %[vw], %[srdw], s48 offen" WNT "\n\t"
        "4:\n\t"
        "s_cmp_lg_u32 %[wv], 0\n\t"
        "s_cbranch_scc1 8f\n\t"
        "s_mov_b32 s50, 4\n\t"
        "s_mov_b32 s41, 0\n\t"
        "6:\n\t"
        "s_memrealtime s[56:57]\n\t"
        "7:\n\t"
        "v_lshlrev_b32 v240, 5, %[l4]\n\t"
        "s_mov_b32 s49, 4194304\n\t"
        "s_bfm_b64 exec, 8, s41\n\t"
        "buffer_load_dwordx4 v[240:243], v240, %[srda], s49 offen sc0 sc1\n\t"
        "s_mov_b64 exec, -1\n\t"
        "s_memrealtime s[54:55]\n\t"
        "s_add_u32 s53, s53, 1\n\t"
        "s_waitcnt vmcnt(0) lgkmcnt(0)\n\t"
        "s_bfm_b64 exec, 8, s41\n\t"
        "v_min3_u32 v240, v240, v241, v242\n\t"
        "v_min_u32 v240, v240, v243\n\t"
        "v_cmp_gt_u32 vcc, s43, v240\n\t"
        "s_mov_b64 exec, -1\n\t"
        "s_add_u32 s41, s41, 8\n\t"
        "s_ff1_i32_b64 s49, vcc\n\t"
        "s_min_u32 s41, s41, s49\n\t"
        "s_min_u32 s41, s41, 64\n\t"
        "s_cmp_ge_u32 s41, s50\n\t"
        "s_cbranch_scc1 8f\n\t"
        "s_sub_u32 s49, s54, s56\n\t"
        "s_sleep 1\n\t"
        "s_cmp_lt_u32 s49, 50000000\n\t"
        "s_cbranch_scc1 7b\n\t"
        "v_mov_b32 v240, 0\n\t"
        "v_mov_b32 v241, s43\n\t"
        "global_store_dword v240, v241, %[tmo] sc1\n\t"
        "ds_write_b32 v240, v241 offset:2624\n\t"
        "s_waitcnt vmcnt(0) lgkmcnt(0)\n\t"
        "s_mov_b32 s41, 64\n\t"
        "s_branch 8f\n\t"
        "8:\n\t"
        "s_waitcnt lgkmcnt(0)\n\t"
        "s_barrier\n\t"
        "s_mov_b32 s49, 4202496\n\t"
        "buffer_load_dword %[pl], %[l4], %[srda], s49 offen sc1\n\t"
        "s_mov_b32 s40, 0\n\t"
        "s_lshl_b32 s58, %[wv], 12\n\t"
        "s_sub_u32 s58, 32768, s58\n\t"
        "v_add_u32 %[ab], s58, %[vx]\n\t"
        "ds_read_b128 v[192:195], %[ab]\n\t"
        "ds_read_b128 v[196:199], %[ab] offset:1024\n\t"
        "s_cmp_lt_u32 %[wv], 4\n\t"
        "s_cbranch_scc0 9f\n\t"
        "s_add_u32 s50, s40, 2\n\t"
        "s_bitcmp1_b64 s[44:45], s50\n\t"
        "s_cselect_b32 s49, 0x8000, 0\n\t"
        "s_lshl_b32 s50, s50, 16\n\t"
        "s_add_u32 s49, s49, s50\n\t"
        "buffer_load_dword %[pd], %[vp], %[srda], s49 offen sc0 sc1\n\t"
        "s_bitcmp1_b64 s[44:45], s40\n\t"
        "s_cselect_b32 s46, 0x8000, 0\n\t"
        "s_lshl_b32 s47, s40, 16\n\t"
        "s_add_u32 s46, s46, s47\n\t"
        "s_xor_b32 s47, s46, 0x800\n\t"
        "buffer_load_dwordx4 v[128:131], %[vx], %[srda], s46 offen offset:0 sc0 sc1\n\t"
        "buffer_load_dwordx4 v[132:135], %[vx], %[srda], s46 offen offset:1024 sc0 sc1\n\t"
        "buffer_load_dwordx4 v[136:139], %[vx], %[srda], s47 offen offset:0 sc0 sc1\n\t"
        "buffer_load_dwordx4 v[140:143], %[vx], %[srda], s47 offen offset:1024 sc0 sc1\n\t"
        "s_add_u32 s40, s40, 1\n\t"
        "s_bitcmp1_b64 s[44:45], s40\n\t"
        "s_cselect_b32 s46, 0x8000, 0\n\t"
        "s_lshl_b32 s47, s40, 16\n\t"
        "s_add_u32 s46, s46, s47\n\t"
        "s_xor_b32 s47, s46, 0x800\n\t"
        "buffer_load_dwordx4 v[144:147], %[vx], %[srda], s46 offen offset:0 sc0 sc1\n\t"
        "buffer_load_dwordx4 v[148:151], %[vx], %[srda], s46 offen offset:1024 sc0 sc1\n\t"
        "buffer_load_dwordx4 v[152:155], %[vx], %[srda], s47 offen offset:0 sc0 sc1\n\t"
        "buffer_load_dwordx4 v[156:159], %[vx], %[srda], s47 offen offset:1024 sc0 sc1\n\t"
        "s_add_u32 s40, s40, 1\n\t"
        "s_bitcmp1_b64 s[44:45], s40\n\t"
        "s_cselect_b32 s46, 0x8000, 0\n\t"
        "s_lshl_b32 s47, s40, 16\n\t"
        "s_add_u32 s46, s46, s47\n\t"
        "s_xor_b32 s47, s46, 0x800\n\t"
        "s_waitcnt vmcnt(10)\n\t"
        "v_cvt_pk_bf16_f32 v240, v208, v212\n\t"
        "v_cvt_pk_bf16_f32 v241, v216, v220\n\t"
        "v_cvt_pk_bf16_f32 v242, v224, v228\n\t"
        "v_cvt_pk_bf16_f32 v243, v232, v236\n\t"
        "ds_write_b128 %[wa], v[240:243] offset:0\n\t"
        "v_cvt_pk_bf16_f32 v240, v209, v213\n\t"
        "v_cvt_pk_bf16_f32 v241, v217, v221\n\t"
        "v_cvt_pk_bf16_f32 v242, v225, v229\n\t"
        "v_cvt_pk_bf16_f32 v243, v233, v237\n\t"
        "ds_write_b128 %[wa], v[240:243] offset:1024\n\t"
        "v_cvt_pk_bf16_f32 v240, v210, v214\n\t"
        "v_cvt_pk_bf16_f32 v241, v218, v222\n\t"
        "v_cvt_pk_bf16_f32 v242, v226, v230\n\t"
        "v_cvt_pk_bf16_f32 v243, v234, v238\n\t"
        "ds_write_b128 %[wa], v[240:243] offset:2048\n\t"
        "v_cvt_pk_bf16_f32 v240, v211, v215\n\t"
        "v_cvt_pk_bf16_f32 v241, v219, v223\n\t"
        "v_cvt_pk_bf16_f32 v242, v227, v231\n\t"
        "v_cvt_pk_bf16_f32 v243, v235, v239\n\t"
        "ds_write_b128 %[wa], v[240:243] offset:3072\n\t"
        "v_xor_b32 %[wa], 0x18000, %[wa]\n\t"
        "s_waitcnt vmcnt(9)\n\t"
        "s_memrealtime s[56:57]\n\t"
        "s_and_b32 s50, s43, 63\n\t"
        "s_mul_i32 s50, s50, 0x04040404\n\t"
        "552:\n\t"
        "v_xor_b32 v240, s50, %[pl]\n\t"
        "v_and_b32 v240, 0xfcfcfcfc, v240\n\t"
        "v_cmp_ne_u32 vcc, 0, v240\n\t"
        "s_cbranch_vccz 553f\n\t"
        "s_mov_b32 s49, 4202496\n\t"
        "s_sleep 2\n\t"
        "buffer_load_dword %[pl], %[l4], %[srda], s49 offen sc1\n\t"
        "s_memrealtime s[54:55]\n\t"
        "s_add_u32 s53, s53, 1\n\t"
        "s_waitcnt vmcnt(0) lgkmcnt(0)\n\t"
        "s_sub_u32 s49, s54, s56\n\t"
        "s_cmp_lt_u32 s49, 50000000\n\t"
        "s_cbranch_scc1 552b\n\t"
        "v_mov_b32 v240, 0\n\t"
        "v_mov_b32 v241, s43\n\t"
        "global_store_dword v240, v241, %[tmo] sc1\n\t"
        "ds_write_b32 v240, v241 offset:2624\n\t"
        "s_waitcnt vmcnt(0) lgkmcnt(0)\n\t"
        "553:\n\t"
        "s_mov_b32 s50, 0\n\t"
        "v_and_b32 v240, 1, %[pl]\n\t"
        "v_cmp_ne_u32 vcc, 0, v240\n\t"
        "s_nop 0\n\t"
        "s_bcnt1_i32_b64 s49, vcc\n\t"
        "s_add_u32 s50, s50, s49\n\t"
        "v_and_b32 v240, 256, %[pl]\n\t"
        "v_cmp_ne_u32 vcc, 0, v240\n\t"
        "s_nop 0\n\t"
        "s_bcnt1_i32_b64 s49, vcc\n\t"
        "s_add_u32 s50, s50, s49\n\t"
        "v_and_b32 v240, 65536, %[pl]\n\t"
        "v_cmp_ne_u32 vcc, 0, v240\n\t"
        "s_nop 0\n\t"
        "s_bcnt1_i32_b64 s49, vcc\n\t"
        "s_add_u32 s50, s50, s49\n\t"
        "v_and_b32 v240, 16777216, %[pl]\n\t"
        "v_cmp_ne_u32 vcc, 0, v240\n\t"
        "s_nop 0\n\t"
        "s_bcnt1_i32_b64 s49, vcc\n\t"
        "s_add_u32 s50, s50, s49\n\t"
        "s_lshr_b32 s49, %[ld], 5\n\t"
        "s_waitcnt lgkmcnt(0)\n\t"
        "v_readlane_b32 s49, %[pl], s49\n\t"
        "s_cmp_gt_u32 s50, 128\n\t"
        "s_cselect_b32 s50, 1, 0\n\t"
        "s_lshr_b32 s49, s49, %[ld]\n\t"
        "s_xor_b32 s49, s49, s50\n\t"
        "s_xor_b32 s49, s49, s44\n\t"
        "s_bitcmp1_b32 s49, 0\n\t"
        "s_cselect_b64 s[54:55], -1, 0\n\t"
        "s_lshl_b64 s[54:55], s[54:55], 8\n\t"
        "s_and_b32 s44, s44, 255\n\t"
        "s_or_b32 s44, s44, s54\n\t"
        "s_mov_b32 s45, s55\n\t"
        "s_waitcnt vmcnt(0)\n\t"
        "s_branch 10f\n\t"
        "9:\n\t"
        "s_add_u32 s50, s40, 4\n\t"
        "s_bitcmp1_b64 s[44:45], s50\n\t"
        "s_cselect_b32 s49, 0x8000, 0\n\t"
        "s_lshl_b32 s50, s50, 16\n\t"
        "s_add_u32 s49, s49, s50\n\t"
        "buffer_load_dword %[pd], %[vp], %[srda], s49 offen sc0 sc1\n\t"
        "s_bitcmp1_b64 s[44:45], s40\n\t"
        "s_cselect_b32 s46, 0x8000, 0\n\t"
        "s_lshl_b32 s47, s40, 16\n\t"
        "s_add_u32 s46, s46, s47\n\t"
        "s_xor_b32 s47, s46, 0x800\n\t"
        "buffer_load_dwordx4 v[160:163], %[vx], %[srda], s46 offen offset:0 sc0 sc1\n\t"
        "buffer_load_dwordx4 v[164:167], %[vx], %[srda], s46 offen offset:1024 sc0 sc1\n\t"
        "buffer_load_dwordx4 v[168:171], %[vx], %[srda], s47 offen offset:0 sc0 sc1\n\t"
        "buffer_load_dwordx4 v[172:175], %[vx], %[srda], s47 offen offset:1024 sc0 sc1\n\t"
        "s_add_u32 s40, s40, 1\n\t"
        "s_bitcmp1_b64 s[44:45], s40\n\t"
        "s_cselect_b32 s46, 0x8000, 0\n\t"
        "s_lshl_b32 s47, s40, 16\n\t"
        "s_add_u32 s46, s46, s47\n\t"
        "s_xor_b32 s47, s46, 0x800\n\t"
        "buffer_load_dwordx4 v[176:179], %[vx], %[srda], s46 offen offset:0 sc0 sc1\n\t"
        "buffer_load_dwordx4 v[180:183], %[vx], %[srda], s46 offen offset:1024 sc0 sc1\n\t"
        "buffer_load_dwordx4 v[184:187], %[vx], %[srda], s47 offen offset:0 sc0 sc1\n\t"
        "buffer_load_dwordx4 v[188:191], %[vx], %[srda], s47 offen offset:1024 sc0 sc1\n\t"
        "s_add_u32 s40, s40, 1\n\t"
        "s_bitcmp1_b64 s[44:45], s40\n\t"
        "s_cselect_b32 s46, 0x8000, 0\n\t"
        "s_lshl_b32 s47, s40, 16\n\t"
        "s_add_u32 s46, s46, s47\n\t"
        "s_xor_b32 s47, s46, 0x800\n\t"
        "s_waitcnt vmcnt(9)\n\t"
        "s_memrealtime s[56:57]\n\t"
        "s_and_b32 s50, s43, 63\n\t"
        "s_mul_i32 s50, s50, 0x04040404\n\t"
        "652:\n\t"
        "v_xor_b32 v240, s50, %[pl]\n\t"
        "v_and_b32 v240, 0xfcfcfcfc, v240\n\t"
        "v_cmp_ne_u32 vcc, 0, v240\n\t"
        "s_cbranch_vccz 653f\n\t"
        "s_mov_b32 s49, 4202496\n\t"
        "s_sleep 2\n\t"
        "buffer_load_dword %[pl], %[l4], %[srda], s49 offen sc1\n\t"
        "s_memrealtime s[54:55]\n\t"
        "s_add_u32 s53, s53, 1\n\t"
        "s_waitcnt vmcnt(0) lgkmcnt(0)\n\t"
        "s_sub_u32 s49, s54, s56\n\t"
        "s_cmp_lt_u32 s49, 50000000\n\t"
        "s_cbranch_scc1 652b\n\t"
        "v_mov_b32 v240, 0\n\t"
        "v_mov_b32 v241, s43\n\t"
        "global_store_dword v240, v241, %[tmo] sc1\n\t"
        "ds_write_b32 v240, v241 offset:2624\n\t"
        "s_waitcnt vmcnt(0) lgkmcnt(0)\n\t"
        "653:\n\t"
        "s_mov_b32 s50, 0\n\t"
        "v_and_b32 v240, 1, %[pl]\n\t"
        "v_cmp_ne_u32 vcc, 0, v240\n\t"
        "s_nop 0\n\t"
        "s_bcnt1_i32_b64 s49, vcc\n\t"
        "s_add_u32 s50, s50, s49\n\t"
        "v_and_b32 v240, 256, %[pl]\n\t"
        "v_cmp_ne_u32 vcc, 0, v240\n\t"
        "s_nop 0\n\t"
        "s_bcnt1_i32_b64 s49, vcc\n\t"
        "s_add_u32 s50, s50, s49\n\t"
        "v_and_b32 v240, 65536, %[pl]\n\t"
        "v_cmp_ne_u32 vcc, 0, v240\n\t"
        "s_nop 0\n\t"
        "s_bcnt1_i32_b64 s49, vcc\n\t"
        "s_add_u32 s50, s50, s49\n\t"
        "v_and_b32 v240, 16777216, %[pl]\n\t"
        "v_cmp_ne_u32 vcc, 0, v240\n\t"
        "s_nop 0\n\t"
        "s_bcnt1_i32_b64 s49, vcc\n\t"
        "s_add_u32 s50, s50, s49\n\t"
        "s_lshr_b32 s49, %[ld], 5\n\t"
        "s_waitcnt lgkmcnt(0)\n\t"
        "v_readlane_b32 s49, %[pl], s49\n\t"
        "s_cmp_gt_u32 s50, 128\n\t"
        "s_cselect_b32 s50, 1, 0\n\t"
        "s_lshr_b32 s49, s49, %[ld]\n\t"
        "s_xor_b32 s49, s49, s50\n\t"
        "s_xor_b32 s49, s49, s44\n\t"
        "s_bitcmp1_b32 s49, 0\n\t"
        "s_cselect_b64 s[54:55], -1, 0\n\t"
        "s_lshl_b64 s[54:55], s[54:55], 8\n\t"
        "s_and_b32 s44, s44, 255\n\t"
        "s_or_b32 s44, s44, s54\n\t"
        "s_mov_b32 s45, s55\n\t"
        "s_waitcnt vmcnt(0)\n\t"
        "s_branch 11f\n\t"
        "10:\n\t"
        "ds_read_b128 v[200:203], %[ab] offset:2048\n\t"
        "ds_read_b128 v[204:207], %[ab] offset:3072\n\t"
        "s_waitcnt lgkmcnt(3)\n\t"
        "v_mfma_f32_16x16x32_bf16 v[0:3], v[192:195], v[128:131], v[0:3]\n\t"
        "s_setprio 0\n\t"
        "s_waitcnt lgkmcnt(2)\n\t"
        "v_mfma_f32_16x16x32_bf16 v[4:7], v[196:199], v[128:131], v[4:7]\n\t"
        "buffer_load_dwordx4 v[160:163], %[vx], %[srda], s46 offen offset:0 sc0 sc1\n\t"
        "v_mfma_f32_16x16x32_bf16 v[32:35], v[192:195], v[132:135], v[32:35]\n\t"
        "v_mfma_f32_16x16x32_bf16 v[36:39], v[196:199], v[132:135], v[36:39]\n\t"
        "v_mfma_f32_16x16x32_bf16 v[64:67], v[192:195], v[136:139], v[64:67]\n\t"
        "v_mfma_f32_16x16x32_bf16 v[68:71], v[196:199], v[136:139], v[68:71]\n\t"
        "buffer_load_dwordx4 v[164:167], %[vx], %[srda], s46 offen offset:1024 sc0 sc1\n\t"
        "v_mfma_f32_16x16x32_bf16 v[96:99], v[192:195], v[140:143], v[96:99]\n\t"
        "v_mfma_f32_16x16x32_bf16 v[100:103], v[196:199], v[140:143], v[100:103]\n\t"
        "ds_read_b128 v[192:195], %[ab] offset:4096\n\t"
        "ds_read_b128 v[196:199], %[ab] offset:5120\n\t"
        "s_waitcnt lgkmcnt(3)\n\t"
        "v_mfma_f32_16x16x32_bf16 v[8:11], v[200:203], v[128:131], v[8:11]\n\t"
        "s_waitcnt lgkmcnt(2)\n\t"
        "v_mfma_f32_16x16x32_bf16 v[12:15], v[204:207], v[128:131], v[12:15]\n\t"
        "buffer_load_dwordx4 v[168:171], %[vx], %[srda], s47 offen offset:0 sc0 sc1\n\t"
        "v_mfma_f32_16x16x32_bf16 v[40:43], v[200:203], v[132:135], v[40:43]\n\t"
        "v_mfma_f32_16x16x32_bf16 v[44:47], v[204:207], v[132:135], v[44:47]\n\t"
        "v_mfma_f32_16x16x32_bf16 v[72:75], v[200:203], v[136:139], v[72:75]\n\t"
        "v_mfma_f32_16x16x32_bf16 v[76:79], v[204:207], v[136:139], v[76:79]\n\t"
        "buffer_load_dwordx4 v[172:175], %[vx], %[srda], s47 offen offset:1024 sc0 sc1\n\t"
        "v_mfma_f32_16x16x32_bf16 v[104:107], v[200:203], v[140:143], v[104:107]\n\t"
        "s_add_u32 s50, s40, 1\n\t"
        "s_bitcmp1_b64 s[44:45], s50\n\t"
        "s_cselect_b32 s46, 0x8000, 0\n\t"
        "s_lshl_b32 s47, s50, 16\n\t"
        "s_add_u32 s46, s46, s47\n\t"
        "s_xor_b32 s47, s46, 0x800\n\t"
        "v_mfma_f32_16x16x32_bf16 v[108:111], v[204:207], v[140:143], v[108:111]\n\t"
        "ds_read_b128 v[200:203], %[ab] offset:6144\n\t"
        "ds_read_b128 v[204:207], %[ab] offset:7168\n\t"
        "s_waitcnt lgkmcnt(3)\n\t"
        "v_mfma_f32_16x16x32_bf16 v[16:19], v[192:195], v[128:131], v[16:19]\n\t"
        "s_waitcnt lgkmcnt(2)\n\t"
        "v_mfma_f32_16x16x32_bf16 v[20:23], v[196:199], v[128:131], v[20:23]\n\t"
        "buffer_load_dwordx4 v[176:179], %[vx], %[srda], s46 offen offset:0 sc0 sc1\n\t"
        "v_mfma_f32_16x16x32_bf16 v[48:51], v[192:195], v[132:135], v[48:51]\n\t"
        "v_mfma_f32_16x16x32_bf16 v[52:55], v[196:199], v[132:135], v[52:55]\n\t"
        "v_mfma_f32_16x16x32_bf16 v[80:83], v[192:195], v[136:139], v[80:83]\n\t"
        "v_mfma_f32_16x16x32_bf16 v[84:87], v[196:199], v[136:139], v[84:87]\n\t"
        "buffer_load_dwordx4 v[180:183], %[vx], %[srda], s46 offen offset:1024 sc0 sc1\n\t"
        "v_mfma_f32_16x16x32_bf16 v[112:115], v[192:195], v[140:143], v[112:115]\n\t"
        "v_mfma_f32_16x16x32_bf16 v[116:119], v[196:199], v[140:143], v[116:119]\n\t"
        "ds_read_b128 v[192:195], %[ab] offset:8192\n\t"
        "ds_read_b128 v[196:199], %[ab] offset:9216\n\t"
        "s_waitcnt lgkmcnt(3)\n\t"
        "v_mfma_f32_16x16x32_bf16 v[24:27], v[200:203], v[128:131], v[24:27]\n\t"
        "s_waitcnt lgkmcnt(2)\n\t"
        "v_mfma_f32_16x16x32_bf16 v[28:31], v[204:207], v[128:131], v[28:31]\n\t"
        "buffer_load_dwordx4 v[184:187], %[vx], %[srda], s47 offen offset:0 sc0 sc1\n\t"
        "v_mfma_f32_16x16x32_bf16 v[56:59], v[200:203], v[132:135], v[56:59]\n\t"
        "v_mfma_f32_16x16x32_bf16 v[60:63], v[204:207], v[132:135], v[60:63]\n\t"
        "v_mfma_f32_16x16x32_bf16 v[88:91], v[200:203], v[136:139], v[88:91]\n\t"
        "v_mfma_f32_16x16x32_bf16 v[92:95], v[204:207], v[136:139], v[92:95]\n\t"
        "buffer_load_dwordx4 v[188:191], %[vx], %[srda], s47 offen offset:1024 sc0 sc1\n\t"
        "v_mfma_f32_16x16x32_bf16 v[120:123], v[200:203], v[140:143], v[120:123]\n\t"
        "s_bitcmp1_b64 %[ev], s40\n\t"
        "s_cbranch_scc1 110f\n\t"
        "111:\n\t"
        "v_mfma_f32_16x16x32_bf16 v[124:127], v[204:207], v[140:143], v[124:127]\n\t"
        "s_add_u32 s50, s40, 4\n\t"
        "s_bitcmp1_b64 s[44:45], s50\n\t"
        "s_cselect_b32 s49, 0x8000, 0\n\t"
        "s_lshl_b32 s50, s50, 16\n\t"
        "s_add_u32 s49, s49, s50\n\t"
        "buffer_load_dword %[pd], %[vp], %[srda], s49 offen sc0 sc1\n\t"
        "s_add_u32 s50, s40, %[s4]\n\t"
        "v_readlane_b32 s48, %[kt], s50\n\t"
        "s_bitcmp1_b64 s[44:45], s50\n\t"
        "s_cselect_b32 s49, 0x200000, 0\n\t"
        "s_add_u32 s48, s48, s49\n\t"
        "s_cmp_lt_u32 s50, 64\n\t"
        "s_cselect_b32 s48, s48, 0x80000000\n\t"
        "ds_read_b128 v[200:203], %[ab] offset:10240\n\t"
        "ds_read_b128 v[204:207], %[ab] offset:11264\n\t"
        "s_waitcnt lgkmcnt(3)\n\t"
        "v_mfma_f32_16x16x32_bf16 v[0:3], v[192:195], v[144:147], v[0:3]\n\t"
        "s_setprio 1\n\t"
        "buffer_load_dwordx4 v[208:211], %[vw], %[srdw], s48 offen" WNT "\n\t"
        "s_add_u32 s48, s48, 0x10000\n\t"
        "s_waitcnt lgkmcnt(2)\n\t"
        "v_mfma_f32_16x16x32_bf16 v[4:7], v[196:199], v[144:147], v[4:7]\n\t"
        "v_mfma_f32_16x16x32_bf16 v[32:35], v[192:195], v[148:151], v[32:35]\n\t"
        "buffer_load_dwordx4 v[212:215], %[vw], %[srdw], s48 offen" WNT "\n\t"
        "s_add_u32 s48, s48, 0x10000\n\t"
        "v_mfma_f32_16x16x32_bf16 v[36:39], v[196:199], v[148:151], v[36:39]\n\t"
        "v_mfma_f32_16x16x32_bf16 v[64:67], v[192:195], v[152:155], v[64:67]\n\t"
        "buffer_load_dwordx4 v[216:219], %[vw], %[srdw], s48 offen" WNT "\n\t"
        "s_add_u32 s48, s48, 0x10000\n\t"
        "v_mfma_f32_16x16x32_bf16 v[68:71], v[196:199], v[152:155], v[68:71]\n\t"
        "v_mfma_f32_16x16x32_bf16 v[96:99], v[192:195], v[156:159], v[96:99]\n\t"
        "buffer_load_dwordx4 v[220:223], %[vw], %[srdw], s48 offen" WNT "\n\t"
        "s_add_u32 s48, s48, 0x10000\n\t"
        "v_mfma_f32_16x16x32_bf16 v[100:103], v[196:199], v[156:159], v[100:103]\n\t"
        "ds_read_b128 v[192:195], %[ab] offset:12288\n\t"
        "ds_read_b128 v[196:199], %[ab] offset:13312\n\t"
        "s_waitcnt lgkmcnt(3)\n\t"
        "v_mfma_f32_16x16x32_bf16 v[8:11], v[200:203], v[144:147], v[8:11]\n\t"
        "buffer_load_dwordx4 v[224:227], %[vw], %[srdw], s48 offen" WNT "\n\t"
        "s_add_u32 s48, s48, 0x10000\n\t"
        "s_waitcnt lgkmcnt(2)\n\t"
        "v_mfma_f32_16x16x32_bf16 v[12:15], v[204:207], v[144:147], v[12:15]\n\t"
        "v_mfma_f32_16x16x32_bf16 v[40:43], v[200:203], v[148:151], v[40:43]\n\t"
        "buffer_load_dwordx4 v[228:231], %[vw], %[srdw], s48 offen" WNT "\n\t"
        "s_add_u32 s48, s48, 0x10000\n\t"
        "v_mfma_f32_16x16x32_bf16 v[44:47], v[204:207], v[148:151], v[44:47]\n\t"
        "v_mfma_f32_16x16x32_bf16 v[72:75], v[200:203], v[152:155], v[72:75]\n\t"
        "buffer_load_dwordx4 v[232:235], %[vw], %[srdw], s48 offen" WNT "\n\t"
        "s_add_u32 s48, s48, 0x10000\n\t"
        "v_mfma_f32_16x16x32_bf16 v[76:79], v[204:207], v[152:155], v[76:79]\n\t"
        "v_mfma_f32_16x16x32_bf16 v[104:107], v[200:203], v[156:159], v[104:107]\n\t"
        "buffer_load_dwordx4 v[236:239], %[vw], %[srdw], s48 offen" WNT "\n\t"
        "v_mfma_f32_16x16x32_bf16 v[108:111], v[204:207], v[156:159], v[108:111]\n\t"
        "ds_read_b128 v[200:203], %[ab] offset:14336\n\t"
        "ds_read_b128 v[204:207], %[ab] offset:15360\n\t"
        "s_waitcnt lgkmcnt(3)\n\t"
        "v_mfma_f32_16x16x32_bf16 v[16:19], v[192:195], v[144:147], v[16:19]\n\t"
        "s_waitcnt lgkmcnt(2)\n\t"
        "v_mfma_f32_16x16x32_bf16 v[20:23], v[196:199], v[144:147], v[20:23]\n\t"
        "v_mfma_f32_16x16x32_bf16 v[48:51], v[192:195], v[148:151], v[48:51]\n\t"
        "v_mfma_f32_16x16x32_bf16 v[52:55], v[196:199], v[148:151], v[52:55]\n\t"
        "s_lshl_b32 s49, s40, 13\n\t"
        "s_and_b32 s49, s49, 0xc000\n\t"
        "s_add_u32 s49, s49, s58\n\t"
        "v_add_u32 %[ab], s49, %[vx]\n\t"
        "v_mfma_f32_16x16x32_bf16 v[80:83], v[192:195], v[152:155], v[80:83]\n\t"
        "v_mfma_f32_16x16x32_bf16 v[84:87], v[196:199], v[152:155], v[84:87]\n\t"
        "v_mfma_f32_16x16x32_bf16 v[112:115], v[192:195], v[156:159], v[112:115]\n\t"
        "v_mfma_f32_16x16x32_bf16 v[116:119], v[196:199], v[156:159], v[116:119]\n\t"
        "ds_read_b128 v[192:195], %[ab] offset:0\n\t"
        "ds_read_b128 v[196:199], %[ab] offset:1024\n\t"
        "s_waitcnt lgkmcnt(3)\n\t"
        "v_mfma_f32_16x16x32_bf16 v[24:27], v[200:203], v[144:147], v[24:27]\n\t"
        "s_waitcnt lgkmcnt(2)\n\t"
        "v_mfma_f32_16x16x32_bf16 v[28:31], v[204:207], v[144:147], v[28:31]\n\t"
        "v_mfma_f32_16x16x32_bf16 v[56:59], v[200:203], v[148:151], v[56:59]\n\t"
        "s_waitcnt vmcnt(11)\n\t"
        "s_bitcmp1_b64 %[ev], s40\n\t"
        "s_cbranch_scc1 140f\n\t"
        "s_waitcnt vmcnt(9)\n\t"
        "141:\n\t"
        "v_mfma_f32_16x16x32_bf16 v[60:63], v[204:207], v[148:151], v[60:63]\n\t"
        "v_mfma_f32_16x16x32_bf16 v[88:91], v[200:203], v[152:155], v[88:91]\n\t"
        "s_add_u32 s40, s40, 2\n\t"
        "s_bitcmp1_b64 s[44:45], s40\n\t"
        "s_cselect_b32 s46, 0x8000, 0\n\t"
        "s_lshl_b32 s47, s40, 16\n\t"
        "s_add_u32 s46, s46, s47\n\t"
        "s_xor_b32 s47, s46, 0x800\n\t"
        "v_mfma_f32_16x16x32_bf16 v[92:95], v[204:207], v[152:155], v[92:95]\n\t"
        "v_mfma_f32_16x16x32_bf16 v[120:123], v[200:203], v[156:159], v[120:123]\n\t"
        "v_mfma_f32_16x16x32_bf16 v[124:127], v[204:207], v[156:159], v[124:127]\n\t"
        "s_barrier\n\t"
        "s_cmp_lt_u32 s40, 60\n\t"
        "s_cbranch_scc0 18f\n\t"
        "11:\n\t"
        "ds_read_b128 v[200:203], %[ab] offset:2048\n\t"
        "ds_read_b128 v[204:207], %[ab] offset:3072\n\t"
        "s_waitcnt lgkmcnt(3)\n\t"
        "v_mfma_f32_16x16x32_bf16 v[0:3], v[192:195], v[160:163], v[0:3]\n\t"
        "s_setprio 1\n\t"
        "s_waitcnt lgkmcnt(2)\n\t"
        "v_mfma_f32_16x16x32_bf16 v[4:7], v[196:199], v[160:163], v[4:7]\n\t"
        "buffer_load_dwordx4 v[128:131], %[vx], %[srda], s46 offen offset:0 sc0 sc1\n\t"
        "v_mfma_f32_16x16x32_bf16 v[32:35], v[192:195], v[164:167], v[32:35]\n\t"
        "v_mfma_f32_16x16x32_bf16 v[36:39], v[196:199], v[164:167], v[36:39]\n\t"
        "v_mfma_f32_16x16x32_bf16 v[64:67], v[192:195], v[168:171], v[64:67]\n\t"
        "v_mfma_f32_16x16x32_bf16 v[68:71], v[196:199], v[168:171], v[68:71]\n\t"
        "buffer_load_dwordx4 v[132:135], %[vx], %[srda], s46 offen offset:1024 sc0 sc1\n\t"
        "v_mfma_f32_16x16x32_bf16 v[96:99], v[192:195], v[172:175], v[96:99]\n\t"
        "v_mfma_f32_16x16x32_bf16 v[100:103], v[196:199], v[172:175], v[100:103]\n\t"
        "ds_read_b128 v[192:195], %[ab] offset:4096\n\t"
        "ds_read_b128 v[196:199], %[ab] offset:5120\n\t"
        "s_waitcnt lgkmcnt(3)\n\t"
        "v_mfma_f32_16x16x32_bf16 v[8:11], v[200:203], v[160:163], v[8:11]\n\t"
        "s_waitcnt lgkmcnt(2)\n\t"
        "v_mfma_f32_16x16x32_bf16 v[12:15], v[204:207], v[160:163], v[12:15]\n\t"
        "buffer_load_dwordx4 v[136:139], %[vx], %[srda], s47 offen offset:0 sc0 sc1\n\t"
        "v_mfma_f32_16x16x32_bf16 v[40:43], v[200:203], v[164:167], v[40:43]\n\t"
        "v_mfma_f32_16x16x32_bf16 v[44:47], v[204:207], v[164:167], v[44:47]\n\t"
        "v_mfma_f32_16x16x32_bf16 v[72:75], v[200:203], v[168:171], v[72:75]\n\t"
        "v_mfma_f32_16x16x32_bf16 v[76:79], v[204:207], v[168:171], v[76:79]\n\t"
        "buffer_load_dwordx4 v[140:143], %[vx], %[srda], s47 offen offset:1024 sc0 sc1\n\t"
        "v_mfma_f32_16x16x32_bf16 v[104:107], v[200:203], v[172:175], v[104:107]\n\t"
        "s_add_u32 s50, s40, 1\n\t"
        "s_bitcmp1_b64 s[44:45], s50\n\t"
        "s_cselect_b32 s46, 0x8000, 0\n\t"
        "s_lshl_b32 s47, s50, 16\n\t"
        "s_add_u32 s46, s46, s47\n\t"
        "s_xor_b32 s47, s46, 0x800\n\t"
        "v_mfma_f32_16x16x32_bf16 v[108:111], v[204:207], v[172:175], v[108:111]\n\t"
        "ds_read_b128 v[200:203], %[ab] offset:6144\n\t"
        "ds_read_b128 v[204:207], %[ab] offset:7168\n\t"
        "s_waitcnt lgkmcnt(3)\n\t"
        "v_mfma_f32_16x16x32_bf16 v[16:19], v[192:195], v[160:163], v[16:19]\n\t"
        "s_waitcnt lgkmcnt(2)\n\t"
        "v_mfma_f32_16x16x32_bf16 v[20:23], v[196:199], v[160:163], v[20:23]\n\t"
        "buffer_load_dwordx4 v[144:147], %[vx], %[srda], s46 offen offset:0 sc0 sc1\n\t"
        "v_mfma_f32_16x16x32_bf16 v[48:51], v[192:195], v[164:167], v[48:51]\n\t"
        "v_mfma_f32_16x16x32_bf16 v[52:55], v[196:199], v[164:167], v[52:55]\n\t"
        "v_mfma_f32_16x16x32_bf16 v[80:83], v[192:195], v[168:171], v[80:83]\n\t"
        "v_mfma_f32_16x16x32_bf16 v[84:87], v[196:199], v[168:171], v[84:87]\n\t"
        "buffer_load_dwordx4 v[148:151], %[vx], %[srda], s46 offen offset:1024 sc0 sc1\n\t"
        "v_mfma_f32_16x16x32_bf16 v[112:115], v[192:195], v[172:175], v[112:115]\n\t"
        "v_mfma_f32_16x16x32_bf16 v[116:119], v[196:199], v[172:175], v[116:119]\n\t"
        "ds_read_b128 v[192:195], %[ab] offset:8192\n\t"
        "ds_read_b128 v[196:199], %[ab] offset:9216\n\t"
        "s_waitcnt lgkmcnt(3)\n\t"
        "v_mfma_f32_16x16x32_bf16 v[24:27], v[200:203], v[160:163], v[24:27]\n\t"
        "s_waitcnt lgkmcnt(2)\n\t"
        "v_mfma_f32_16x16x32_bf16 v[28:31], v[204:207], v[160:163], v[28:31]\n\t"
        "buffer_load_dwordx4 v[152:155], %[vx], %[srda], s47 offen offset:0 sc0 sc1\n\t"
        "v_mfma_f32_16x16x32_bf16 v[56:59], v[200:203], v[164:167], v[56:59]\n\t"
        "v_mfma_f32_16x16x32_bf16 v[60:63], v[204:207], v[164:167], v[60:63]\n\t"
        "v_mfma_f32_16x16x32_bf16 v[88:91], v[200:203], v[168:171], v[88:91]\n\t"
        "v_mfma_f32_16x16x32_bf16 v[92:95], v[204:207], v[168:171], v[92:95]\n\t"
        "buffer_load_dwordx4 v[156:159], %[vx], %[srda], s47 offen offset:1024 sc0 sc1\n\t"
        "v_mfma_f32_16x16x32_bf16 v[120:123], v[200:203], v[172:175], v[120:123]\n\t"
        "s_bitcmp1_b64 %[ev], s40\n\t"
        "s_cbranch_scc1 210f\n\t"
        "211:\n\t"
        "v_mfma_f32_16x16x32_bf16 v[124:127], v[204:207], v[172:175], v[124:127]\n\t"
        "s_waitcnt vmcnt(8)\n\t"
        "ds_read_b128 v[200:203], %[ab] offset:10240\n\t"
        "ds_read_b128 v[204:207], %[ab] offset:11264\n\t"
        "s_waitcnt lgkmcnt(3)\n\t"
        "v_mfma_f32_16x16x32_bf16 v[0:3], v[192:195], v[176:179], v[0:3]\n\t"
        "s_setprio 0\n\t"
        "v_cvt_pk_bf16_f32 v240, v208, v212\n\t"
        "s_waitcnt lgkmcnt(2)\n\t"
        "v_mfma_f32_16x16x32_bf16 v[4:7], v[196:199], v[176:179], v[4:7]\n\t"
        "v_cvt_pk_bf16_f32 v241, v216, v220\n\t"
        "v_mfma_f32_16x16x32_bf16 v[32:35], v[192:195], v[180:183], v[32:35]\n\t"
        "v_cvt_pk_bf16_f32 v242, v224, v228\n\t"
        "v_mfma_f32_16x16x32_bf16 v[36:39], v[196:199], v[180:183], v[36:39]\n\t"
        "v_cvt_pk_bf16_f32 v243, v232, v236\n\t"
        "ds_write_b128 %[wa], v[240:243] offset:0\n\t"
        "v_mfma_f32_16x16x32_bf16 v[64:67], v[192:195], v[184:187], v[64:67]\n\t"
        "v_cvt_pk_bf16_f32 v240, v209, v213\n\t"
        "v_mfma_f32_16x16x32_bf16 v[68:71], v[196:199], v[184:187], v[68:71]\n\t"
        "v_cvt_pk_bf16_f32 v241, v217, v221\n\t"
        "v_mfma_f32_16x16x32_bf16 v[96:99], v[192:195], v[188:191], v[96:99]\n\t"
        "v_cvt_pk_bf16_f32 v242, v225, v229\n\t"
        "v_mfma_f32_16x16x32_bf16 v[100:103], v[196:199], v[188:191], v[100:103]\n\t"
        "v_cvt_pk_bf16_f32 v243, v233, v237\n\t"
        "ds_write_b128 %[wa], v[240:243] offset:1024\n\t"
        "ds_read_b128 v[192:195], %[ab] offset:12288\n\t"
        "ds_read_b128 v[196:199], %[ab] offset:13312\n\t"
        "s_waitcnt lgkmcnt(5)\n\t"
        "v_mfma_f32_16x16x32_bf16 v[8:11], v[200:203], v[176:179], v[8:11]\n\t"
        "v_cvt_pk_bf16_f32 v240, v210, v214\n\t"
        "s_waitcnt lgkmcnt(4)\n\t"
        "v_mfma_f32_16x16x32_bf16 v[12:15], v[204:207], v[176:179], v[12:15]\n\t"
        "v_cvt_pk_bf16_f32 v241, v218, v222\n\t"
        "v_mfma_f32_16x16x32_bf16 v[40:43], v[200:203], v[180:183], v[40:43]\n\t"
        "v_cvt_pk_bf16_f32 v242, v226, v230\n\t"
        "v_mfma_f32_16x16x32_bf16 v[44:47], v[204:207], v[180:183], v[44:47]\n\t"
        "v_cvt_pk_bf16_f32 v243, v234, v238\n\t"
        "ds_write_b128 %[wa], v[240:243] offset:2048\n\t"
        "v_mfma_f32_16x16x32_bf16 v[72:75], v[200:203], v[184:187], v[72:75]\n\t"
        "v_cvt_pk_bf16_f32 v240, v211, v215\n\t"
        "v_mfma_f32_16x16x32_bf16 v[76:79], v[204:207], v[184:187], v[76:79]\n\t"
        "v_cvt_pk_bf16_f32 v241, v219, v223\n\t"
        "v_mfma_f32_16x16x32_bf16 v[104:107], v[200:203], v[188:191], v[104:107]\n\t"
        "v_cvt_pk_bf16_f32 v242, v227, v231\n\t"
        "v_mfma_f32_16x16x32_bf16 v[108:111], v[204:207], v[188:191], v[108:111]\n\t"
        "v_cvt_pk_bf16_f32 v243, v235, v239\n\t"
        "ds_write_b128 %[wa], v[240:243] offset:3072\n\t"
        "ds_read_b128 v[200:203], %[ab] offset:14336\n\t"
        "ds_read_b128 v[204:207], %[ab] offset:15360\n\t"
        "s_waitcnt lgkmcnt(5)\n\t"
        "v_mfma_f32_16x16x32_bf16 v[16:19], v[192:195], v[176:179], v[16:19]\n\t"
        "v_xor_b32 %[wa], 0x18000, %[wa]\n\t"
        "s_waitcnt lgkmcnt(4)\n\t"
        "v_mfma_f32_16x16x32_bf16 v[20:23], v[196:199], v[176:179], v[20:23]\n\t"
        "s_cmp_eq_u32 s40, %[tp]\n\t"
        "s_cbranch_scc1 230f\n\t"
        "231:\n\t"
        "v_mfma_f32_16x16x32_bf16 v[48:51], v[192:195], v[180:183], v[48:51]\n\t"
        "v_mfma_f32_16x16x32_bf16 v[52:55], v[196:199], v[180:183], v[52:55]\n\t"
        "s_lshl_b32 s49, s40, 13\n\t"
        "s_and_b32 s49, s49, 0xc000\n\t"
        "s_add_u32 s49, s49, s58\n\t"
        "v_add_u32 %[ab], s49, %[vx]\n\t"
        "v_mfma_f32_16x16x32_bf16 v[80:83], v[192:195], v[184:187], v[80:83]\n\t"
        "v_mfma_f32_16x16x32_bf16 v[84:87], v[196:199], v[184:187], v[84:87]\n\t"
        "v_mfma_f32_16x16x32_bf16 v[112:115], v[192:195], v[188:191], v[112:115]\n\t"
        "v_mfma_f32_16x16x32_bf16 v[116:119], v[196:199], v[188:191], v[116:119]\n\t"
        "ds_read_b128 v[192:195], %[ab] offset:0\n\t"
        "ds_read_b128 v[196:199], %[ab] offset:1024\n\t"
        "s_waitcnt lgkmcnt(3)\n\t"
        "v_mfma_f32_16x16x32_bf16 v[24:27], v[200:203], v[176:179], v[24:27]\n\t"
        "s_waitcnt lgkmcnt(2)\n\t"
        "v_mfma_f32_16x16x32_bf16 v[28:31], v[204:207], v[176:179], v[28:31]\n\t"
        "v_mfma_f32_16x16x32_bf16 v[56:59], v[200:203], v[180:183], v[56:59]\n\t"
        "s_waitcnt vmcnt(2)\n\t"
        "s_bitcmp1_b64 %[ev], s40\n\t"
        "s_cbranch_scc1 240f\n\t"
        "s_waitcnt vmcnt(0)\n\t"
        "241:\n\t"
        "v_mfma_f32_16x16x32_bf16 v[60:63], v[204:207], v[180:183], v[60:63]\n\t"
        "v_mfma_f32_16x16x32_bf16 v[88:91], v[200:203], v[184:187], v[88:91]\n\t"
        "s_add_u32 s40, s40, 2\n\t"
        "s_bitcmp1_b64 s[44:45], s40\n\t"
        "s_cselect_b32 s46, 0x8000, 0\n\t"
        "s_lshl_b32 s47, s40, 16\n\t"
        "s_add_u32 s46, s46, s47\n\t"
        "s_xor_b32 s47, s46, 0x800\n\t"
        "v_mfma_f32_16x16x32_bf16 v[92:95], v[204:207], v[184:187], v[92:95]\n\t"
        "v_mfma_f32_16x16x32_bf16 v[120:123], v[200:203], v[188:191], v[120:123]\n\t"
        "v_mfma_f32_16x16x32_bf16 v[124:127], v[204:207], v[188:191], v[124:127]\n\t"
        "s_barrier\n\t"
        "s_cmp_lt_u32 s40, 60\n\t"
        "s_cbranch_scc1 10b\n\t"
        "s_branch 19f\n\t"
        "110:\n\t"
        "s_cmp_ge_u32 s41, 64\n\t"
        "s_cbranch_scc1 113f\n\t"
        "v_lshlrev_b32 v240, 5, %[l4]\n\t"
        "s_mov_b32 s49, 4194304\n\t"
        "s_bfm_b64 exec, 8, s41\n\t"
        "buffer_load_dwordx4 v[240:243], v240, %[srda], s49 offen sc0 sc1\n\t"
        "s_mov_b64 exec, -1\n\t"
        "113:\n\t"
        "s_cmp_eq_u32 s40, %[tp]\n\t"
        "s_cbranch_scc1 114f\n\t"
        "s_cmp_lg_u32 s40, %[tf]\n\t"
        "s_cbranch_scc1 111b\n\t"
        "v_mov_b32 v208, 0\n\t"
        "v_mov_b32 v209, s43\n\t"
        "s_mov_b64 exec, 1\n\t"
        "global_store_dword v208, v209, %[fl] sc0 sc1\n\t"
        "global_store_dword v208, v209, %[fl] sc0 sc1\n\t"
        "s_mov_b64 exec, -1\n\t"
        "s_mov_b32 s42, 1\n\t"
        "s_branch 111b\n\t"
        "114:\n\t"
        "v_add_u32 v208, 98304, %[vx]\n\t"
        "s_waitcnt vmcnt(8)\n\t"
        "ds_read_b128 v[224:227], v208 offset:0\n\t"
        "ds_read_b128 v[228:231], v208 offset:1024\n\t"
        "ds_read_b128 v[232:235], v208 offset:2048\n\t"
        "ds_read_b128 v[236:239], v208 offset:3072\n\t"
        "s_waitcnt lgkmcnt(0)\n\t"
        "v_cvt_pk_bf16_f32 v208, v224, v225\n\t"
        "v_cvt_pk_bf16_f32 v209, v226, v227\n\t"
        "v_cvt_pk_bf16_f32 v210, v228, v229\n\t"
        "v_cvt_pk_bf16_f32 v211, v230, v231\n\t"
        "v_cvt_pk_bf16_f32 v212, v232, v233\n\t"
        "v_cvt_pk_bf16_f32 v213, v234, v235\n\t"
        "v_cvt_pk_bf16_f32 v214, v236, v237\n\t"
        "v_cvt_pk_bf16_f32 v215, v238, v239\n\t"
        "v_add_u32 v216, 0x8000, %[vx]\n\t"
        "global_store_dwordx4 %[vx], v[208:211], %[pu] sc0 sc1\n\t"
        "global_store_dwordx4 v216, v[212:215], %[pu] sc0 sc1\n\t"
        "s_mov_b32 s42, 1\n\t"
        "s_branch 111b\n\t"
        "140:\n\t"
        "s_cmp_lg_u32 s42, 0\n\t"
        "s_cbranch_scc1 142f\n\t"
        "s_waitcnt vmcnt(9)\n\t"
        "142:\n\t"
        "s_mov_b32 s42, 0\n\t"
        "s_cmp_ge_u32 s41, 64\n\t"
        "s_cbranch_scc1 143f\n\t"
        "s_bfm_b64 exec, 8, s41\n\t"
        "v_min3_u32 v240, v240, v241, v242\n\t"
        "v_min_u32 v240, v240, v243\n\t"
        "v_cmp_gt_u32 vcc, s43, v240\n\t"
        "s_mov_b64 exec, -1\n\t"
        "s_add_u32 s41, s41, 8\n\t"
        "s_ff1_i32_b64 s49, vcc\n\t"
        "s_min_u32 s41, s41, s49\n\t"
        "s_min_u32 s41, s41, 64\n\t"
        "s_add_u32 s50, s40, 4\n\t"
        "s_cmp_ge_u32 s40, 58\n\t"
        "s_cselect_b32 s50, 64, s50\n\t"
        "s_cmp_lt_u32 s41, s50\n\t"
        "s_cbranch_scc1 160f\n\t"
        "143:\n\t"
        "s_cmp_lg_u32 %[wv], 0\n\t"
        "s_cbranch_scc1 145f\n\t"
        "v_readlane_b32 s50, %[dl], s40\n\t"
        "s_cmp_eq_u32 s50, 0\n\t"
        "s_cbranch_scc1 145f\n\t"
        "s_add_u32 s50, s50, s59\n\t"
        "144:\n\t"
        "s_memrealtime s[54:55]\n\t"
        "s_waitcnt lgkmcnt(0)\n\t"
        "s_sub_u32 s49, s54, s50\n\t"
        "s_cmp_lt_i32 s49, 0\n\t"
        "s_cbranch_scc1 144b\n\t"
        "145:\n\t"
        "s_branch 141b\n\t"
        "160:\n\t"
        "s_memrealtime s[56:57]\n\t"
        "161:\n\t"
        "v_lshlrev_b32 v240, 5, %[l4]\n\t"
        "s_mov_b32 s49, 4194304\n\t"
        "s_bfm_b64 exec, 8, s41\n\t"
        "buffer_load_dwordx4 v[240:243], v240, %[srda], s49 offen sc0 sc1\n\t"
        "s_mov_b64 exec, -1\n\t"
        "s_memrealtime s[54:55]\n\t"
        "s_add_u32 s53, s53, 1\n\t"
        "s_waitcnt vmcnt(0) lgkmcnt(0)\n\t"
        "s_bfm_b64 exec, 8, s41\n\t"
        "v_min3_u32 v240, v240, v241, v242\n\t"
        "v_min_u32 v240, v240, v243\n\t"
        "v_cmp_gt_u32 vcc, s43, v240\n\t"
        "s_mov_b64 exec, -1\n\t"
        "s_add_u32 s41, s41, 8\n\t"
        "s_ff1_i32_b64 s49, vcc\n\t"
        "s_min_u32 s41, s41, s49\n\t"
        "s_min_u32 s41, s41, 64\n\t"
        "s_cmp_ge_u32 s41, s50\n\t"
        "s_cbranch_scc1 143b\n\t"
        "s_sub_u32 s49, s54, s56\n\t"
        "s_sleep 1\n\t"
        "s_cmp_lt_u32 s49, 50000000\n\t"
        "s_cbranch_scc1 161b\n\t"
        "v_mov_b32 v240, 0\n\t"
        "v_mov_b32 v241, s43\n\t"
        "global_store_dword v240, v241, %[tmo] sc1\n\t"
        "ds_write_b32 v240, v241 offset:2624\n\t"
        "s_waitcnt vmcnt(0) lgkmcnt(0)\n\t"
        "s_mov_b32 s41, 64\n\t"
        "s_branch 143b\n\t"
        "210:\n\t"
        "s_branch 211b\n\t"
        "230:\n\t"
        "v_add_u32 v208, 98304, %[vx]\n\t"
        "s_waitcnt vmcnt(8)\n\t"
        "ds_read_b128 v[224:227], v208 offset:0\n\t"
        "ds_read_b128 v[228:231], v208 offset:1024\n\t"
        "ds_read_b128 v[232:235], v208 offset:2048\n\t"
        "ds_read_b128 v[236:239], v208 offset:3072\n\t"
        "s_waitcnt lgkmcnt(0)\n\t"
        "v_cvt_pk_bf16_f32 v208, v224, v225\n\t"
        "v_cvt_pk_bf16_f32 v209, v226, v227\n\t"
        "v_cvt_pk_bf16_f32 v210, v228, v229\n\t"
        "v_cvt_pk_bf16_f32 v211, v230, v231\n\t"
        "v_cvt_pk_bf16_f32 v212, v232, v233\n\t"
        "v_cvt_pk_bf16_f32 v213, v234, v235\n\t"
        "v_cvt_pk_bf16_f32 v214, v236, v237\n\t"
        "v_cvt_pk_bf16_f32 v215, v238, v239\n\t"
        "v_add_u32 v216, 0x8000, %[vx]\n\t"
        "global_store_dwordx4 %[vx], v[208:211], %[pu] sc0 sc1\n\t"
        "global_store_dwordx4 v216, v[212:215], %[pu] sc0 sc1\n\t"
        "s_mov_b32 s42, 1\n\t"
        "s_branch 231b\n\t"
        "240:\n\t"
        "s_cmp_lg_u32 s42, 0\n\t"
        "s_cbranch_scc1 242f\n\t"
        "s_waitcnt vmcnt(0)\n\t"
        "242:\n\t"
        "s_mov_b32 s42, 0\n\t"
        "s_cmp_ge_u32 s41, 64\n\t"
        "s_cbranch_scc1 243f\n\t"
        "s_add_u32 s50, s40, 4\n\t"
        "s_cmp_ge_u32 s40, 58\n\t"
        "s_cselect_b32 s50, 64, s50\n\t"
        "s_cmp_lt_u32 s41, s50\n\t"
        "s_cbranch_scc1 260f\n\t"
        "243:\n\t"
        "s_cmp_lg_u32 %[wv], 0\n\t"
        "s_cbranch_scc1 245f\n\t"
        "v_readlane_b32 s50, %[dl], s40\n\t"
        "s_cmp_eq_u32 s50, 0\n\t"
        "s_cbranch_scc1 245f\n\t"
        "s_add_u32 s50, s50, s59\n\t"
        "244:\n\t"
        "s_memrealtime s[54:55]\n\t"
        "s_waitcnt lgkmcnt(0)\n\t"
        "s_sub_u32 s49, s54, s50\n\t"
        "s_cmp_lt_i32 s49, 0\n\t"
        "s_cbranch_scc1 244b\n\t"
        "245:\n\t"
        "s_cmp_lg_u32 s40, %[td]\n\t"
        "s_cbranch_scc1 249f\n\t"
        "v_and_b32 v240, 60, %[l4]\n\t"
        "v_lshrrev_b32 v241, 6, %[l4]\n\t"
        "v_lshlrev_b32 v240, 9, v240\n\t"
        "v_lshl_add_u32 v240, v241, 5, v240\n\t"
        "s_lshl_b32 s49, %[wv], 12\n\t"
        "s_add_u32 m0, s49, 98304\n\t"
        "s_nop 0\n\t"
        "global_load_lds_dwordx4 v240, %[xp] offset:0\n\t"
        "s_add_u32 m0, s49, 99312\n\t"
        "s_nop 0\n\t"
        "global_load_lds_dwordx4 v240, %[xp] offset:16\n\t"
        "s_add_u32 m0, s49, 100224\n\t"
        "s_nop 0\n\t"
        "global_load_lds_dwordx4 v240, %[xp] offset:128\n\t"
        "s_add_u32 m0, s49, 101232\n\t"
        "s_nop 0\n\t"
        "global_load_lds_dwordx4 v240, %[xp] offset:144\n\t"
        "249:\n\t"
        "s_cmp_lg_u32 s40, 52\n\t"
        "s_cbranch_scc1 247f\n\t"
        "s_getpc_b64 s[54:55]\n\t"
        "246:\n\t"
        "s_add_u32 s54, s54, 18f-246b\n\t"
        "s_addc_u32 s55, s55, 0\n\t"
        "v_lshlrev_b32 v240, 5, %[l4]\n\t"
        "global_load_dword %[pd], v240, s[54:55]\n\t"
        "v_add_u32 v240, 0x2000, v240\n\t"
        "v_min_u32 v240, a2a_code_end-18f, v240\n\t"
        "global_load_dword %[pd], v240, s[54:55]\n\t"
        "247:\n\t"
        "s_branch 241b\n\t"
        "260:\n\t"
        "s_memrealtime s[56:57]\n\t"
        "261:\n\t"
        "v_lshlrev_b32 v240, 5, %[l4]\n\t"
        "s_mov_b32 s49, 4194304\n\t"
        "s_bfm_b64 exec, 8, s41\n\t"
        "buffer_load_dwordx4 v[240:243], v240, %[srda], s49 offen sc0 sc1\n\t"
        "s_mov_b64 exec, -1\n\t"
        "s_memrealtime s[54:55]\n\t"
        "s_add_u32 s53, s53, 1\n\t"
        "s_waitcnt vmcnt(0) lgkmcnt(0)\n\t"
        "s_bfm_b64 exec, 8, s41\n\t"
        "v_min3_u32 v240, v240, v241, v242\n\t"
        "v_min_u32 v240, v240, v243\n\t"
        "v_cmp_gt_u32 vcc, s43, v240\n\t"
        "s_mov_b64 exec, -1\n\t"
        "s_add_u32 s41, s41, 8\n\t"
        "s_ff1_i32_b64 s49, vcc\n\t"
        "s_min_u32 s41, s41, s49\n\t"
        "s_min_u32 s41, s41, 64\n\t"
        "s_cmp_ge_u32 s41, s50\n\t"
        "s_cbranch_scc1 243b\n\t"
        "s_sub_u32 s49, s54, s56\n\t"
        "s_sleep 1\n\t"
        "s_cmp_lt_u32 s49, 50000000\n\t"
        "s_cbranch_scc1 261b\n\t"
        "v_mov_b32 v240, 0\n\t"
        "v_mov_b32 v241, s43\n\t"
        "global_store_dword v240, v241, %[tmo] sc1\n\t"
        "ds_write_b32 v240, v241 offset:2624\n\t"
        "s_waitcnt vmcnt(0) lgkmcnt(0)\n\t"
        "s_mov_b32 s41, 64\n\t"
        "s_branch 243b\n\t"
        "18:\n\t"
        "ds_read_b128 v[200:203], %[ab] offset:2048\n\t"
        "ds_read_b128 v[204:207], %[ab] offset:3072\n\t"
        "s_waitcnt lgkmcnt(3)\n\t"
        "v_mfma_f32_16x16x32_bf16 v[0:3], v[192:195], v[160:163], v[0:3]\n\t"
        "buffer_load_dwordx4 v[128:131], %[vx], %[srda], s46 offen offset:0 sc0 sc1\n\t"
        "s_setprio 2\n\t"
        "v_mfma_f32_16x16x32_bf16 v[32:35], v[192:195], v[164:167], v[32:35]\n\t"
        "buffer_load_dwordx4 v[132:135], %[vx], %[srda], s46 offen offset:1024 sc0 sc1\n\t"
        "v_mfma_f32_16x16x32_bf16 v[64:67], v[192:195], v[168:171], v[64:67]\n\t"
        "buffer_load_dwordx4 v[136:139], %[vx], %[srda], s47 offen offset:0 sc0 sc1\n\t"
        "v_mfma_f32_16x16x32_bf16 v[96:99], v[192:195], v[172:175], v[96:99]\n\t"
        "buffer_load_dwordx4 v[140:143], %[vx], %[srda], s47 offen offset:1024 sc0 sc1\n\t"
        "ds_read_b128 v[192:195], %[ab] offset:8192\n\t"
        "s_waitcnt lgkmcnt(3)\n\t"
        "v_mfma_f32_16x16x32_bf16 v[4:7], v[196:199], v[160:163], v[4:7]\n\t"
        "s_mov_b32 s50, 61\n\t"
        "s_bitcmp1_b64 s[44:45], s50\n\t"
        "s_cselect_b32 s46, 0x8000, 0\n\t"
        "s_lshl_b32 s47, s50, 16\n\t"
        "s_add_u32 s46, s46, s47\n\t"
        "s_xor_b32 s47, s46, 0x800\n\t"
        "buffer_load_dwordx4 v[144:147], %[vx], %[srda], s46 offen offset:0 sc0 sc1\n\t"
        "v_mfma_f32_16x16x32_bf16 v[36:39], v[196:199], v[164:167], v[36:39]\n\t"
        "buffer_load_dwordx4 v[148:151], %[vx], %[srda], s46 offen offset:1024 sc0 sc1\n\t"
        "v_mfma_f32_16x16x32_bf16 v[68:71], v[196:199], v[168:171], v[68:71]\n\t"
        "buffer_load_dwordx4 v[152:155], %[vx], %[srda], s47 offen offset:0 sc0 sc1\n\t"
        "v_mfma_f32_16x16x32_bf16 v[100:103], v[196:199], v[172:175], v[100:103]\n\t"
        "buffer_load_dwordx4 v[156:159], %[vx], %[srda], s47 offen offset:1024 sc0 sc1\n\t"
        "ds_read_b128 v[196:199], %[ab] offset:9216\n\t"
        "s_waitcnt lgkmcnt(3)\n\t"
        "v_mfma_f32_16x16x32_bf16 v[8:11], v[200:203], v[160:163], v[8:11]\n\t"
        "s_mov_b32 s50, 62\n\t"
        "s_bitcmp1_b64 s[44:45], s50\n\t"
        "s_cselect_b32 s46, 0x8000, 0\n\t"
        "s_lshl_b32 s47, s50, 16\n\t"
        "s_add_u32 s46, s46, s47\n\t"
        "s_xor_b32 s47, s46, 0x800\n\t"
        "buffer_load_dwordx4 v[208:211], %[vx], %[srda], s46 offen offset:0 sc0 sc1\n\t"
        "v_mfma_f32_16x16x32_bf16 v[40:43], v[200:203], v[164:167], v[40:43]\n\t"
        "buffer_load_dwordx4 v[212:215], %[vx], %[srda], s46 offen offset:1024 sc0 sc1\n\t"
        "v_mfma_f32_16x16x32_bf16 v[72:75], v[200:203], v[168:171], v[72:75]\n\t"
        "buffer_load_dwordx4 v[216:219], %[vx], %[srda], s47 offen offset:0 sc0 sc1\n\t"
        "v_mfma_f32_16x16x32_bf16 v[104:107], v[200:203], v[172:175], v[104:107]\n\t"
        "buffer_load_dwordx4 v[220:223], %[vx], %[srda], s47 offen offset:1024 sc0 sc1\n\t"
        "ds_read_b128 v[200:203], %[ab] offset:10240\n\t"
        "s_waitcnt lgkmcnt(3)\n\t"
        "v_mfma_f32_16x16x32_bf16 v[12:15], v[204:207], v[160:163], v[12:15]\n\t"
        "s_mov_b32 s50, 63\n\t"
        "s_bitcmp1_b64 s[44:45], s50\n\t"
        "s_cselect_b32 s46, 0x8000, 0\n\t"
        "s_lshl_b32 s47, s50, 16\n\t"
        "s_add_u32 s46, s46, s47\n\t"
        "s_xor_b32 s47, s46, 0x800\n\t"
        "buffer_load_dwordx4 v[224:227], %[vx], %[srda], s46 offen offset:0 sc0 sc1\n\t"
        "v_mfma_f32_16x16x32_bf16 v[44:47], v[204:207], v[164:167], v[44:47]\n\t"
        "buffer_load_dwordx4 v[228:231], %[vx], %[srda], s46 offen offset:1024 sc0 sc1\n\t"
        "v_mfma_f32_16x16x32_bf16 v[76:79], v[204:207], v[168:171], v[76:79]\n\t"
        "buffer_load_dwordx4 v[232:235], %[vx], %[srda], s47 offen offset:0 sc0 sc1\n\t"
        "v_mfma_f32_16x16x32_bf16 v[108:111], v[204:207], v[172:175], v[108:111]\n\t"
        "buffer_load_dwordx4 v[236:239], %[vx], %[srda], s47 offen offset:1024 sc0 sc1\n\t"
        "ds_read_b128 v[204:207], %[ab] offset:11264\n\t"
        "s_waitcnt lgkmcnt(3)\n\t"
        "v_mfma_f32_16x16x32_bf16 v[0:3], v[192:195], v[176:179], v[0:3]\n\t"
        "v_mfma_f32_16x16x32_bf16 v[32:35], v[192:195], v[180:183], v[32:35]\n\t"
        "v_mfma_f32_16x16x32_bf16 v[64:67], v[192:195], v[184:187], v[64:67]\n\t"
        "v_mfma_f32_16x16x32_bf16 v[96:99], v[192:195], v[188:191], v[96:99]\n\t"
        "ds_read_b128 v[192:195], %[ab] offset:16384\n\t"
        "s_waitcnt lgkmcnt(3)\n\t"
        "v_mfma_f32_16x16x32_bf16 v[4:7], v[196:199], v[176:179], v[4:7]\n\t"
        "v_mfma_f32_16x16x32_bf16 v[36:39], v[196:199], v[180:183], v[36:39]\n\t"
        "v_mfma_f32_16x16x32_bf16 v[68:71], v[196:199], v[184:187], v[68:71]\n\t"
        "v_mfma_f32_16x16x32_bf16 v[100:103], v[196:199], v[188:191], v[100:103]\n\t"
        "ds_read_b128 v[196:199], %[ab] offset:17408\n\t"
        "s_waitcnt lgkmcnt(3)\n\t"
        "v_mfma_f32_16x16x32_bf16 v[8:11], v[200:203], v[176:179], v[8:11]\n\t"
        "v_mfma_f32_16x16x32_bf16 v[40:43], v[200:203], v[180:183], v[40:43]\n\t"
        "v_mfma_f32_16x16x32_bf16 v[72:75], v[200:203], v[184:187], v[72:75]\n\t"
        "v_mfma_f32_16x16x32_bf16 v[104:107], v[200:203], v[188:191], v[104:107]\n\t"
        "ds_read_b128 v[200:203], %[ab] offset:24576\n\t"
        "s_waitcnt lgkmcnt(3)\n\t"
        "v_mfma_f32_16x16x32_bf16 v[12:15], v[204:207], v[176:179], v[12:15]\n\t"
        "v_mfma_f32_16x16x32_bf16 v[44:47], v[204:207], v[180:183], v[44:47]\n\t"
        "v_mfma_f32_16x16x32_bf16 v[76:79], v[204:207], v[184:187], v[76:79]\n\t"
        "v_mfma_f32_16x16x32_bf16 v[108:111], v[204:207], v[188:191], v[108:111]\n\t"
        "ds_read_b128 v[204:207], %[ab] offset:25600\n\t"
        "s_waitcnt lgkmcnt(3)\n\t"
        "s_waitcnt vmcnt(15)\n\t"
        "v_mfma_f32_16x16x32_bf16 v[0:3], v[192:195], v[128:131], v[0:3]\n\t"
        "s_waitcnt vmcnt(14)\n\t"
        "v_mfma_f32_16x16x32_bf16 v[32:35], v[192:195], v[132:135], v[32:35]\n\t"
        "s_waitcnt vmcnt(13)\n\t"
        "v_mfma_f32_16x16x32_bf16 v[64:67], v[192:195], v[136:139], v[64:67]\n\t"
        "s_waitcnt vmcnt(12)\n\t"
        "v_mfma_f32_16x16x32_bf16 v[96:99], v[192:195], v[140:143], v[96:99]\n\t"
        "ds_read_b128 v[192:195], %[ab] offset:32768\n\t"
        "s_waitcnt lgkmcnt(3)\n\t"
        "v_mfma_f32_16x16x32_bf16 v[4:7], v[196:199], v[128:131], v[4:7]\n\t"
        "v_mfma_f32_16x16x32_bf16 v[36:39], v[196:199], v[132:135], v[36:39]\n\t"
        "v_mfma_f32_16x16x32_bf16 v[68:71], v[196:199], v[136:139], v[68:71]\n\t"
        "v_mfma_f32_16x16x32_bf16 v[100:103], v[196:199], v[140:143], v[100:103]\n\t"
        "ds_read_b128 v[196:199], %[ab] offset:33792\n\t"
        "s_waitcnt lgkmcnt(3)\n\t"
        "s_waitcnt vmcnt(11)\n\t"
        "v_mfma_f32_16x16x32_bf16 v[0:3], v[200:203], v[144:147], v[0:3]\n\t"
        "s_waitcnt vmcnt(10)\n\t"
        "v_mfma_f32_16x16x32_bf16 v[32:35], v[200:203], v[148:151], v[32:35]\n\t"
        "s_waitcnt vmcnt(9)\n\t"
        "v_mfma_f32_16x16x32_bf16 v[64:67], v[200:203], v[152:155], v[64:67]\n\t"
        "s_waitcnt vmcnt(8)\n\t"
        "v_mfma_f32_16x16x32_bf16 v[96:99], v[200:203], v[156:159], v[96:99]\n\t"
        "ds_read_b128 v[200:203], %[ab] offset:40960\n\t"
        "s_waitcnt lgkmcnt(3)\n\t"
        "v_mfma_f32_16x16x32_bf16 v[4:7], v[204:207], v[144:147], v[4:7]\n\t"
        "v_mfma_f32_16x16x32_bf16 v[36:39], v[204:207], v[148:151], v[36:39]\n\t"
        "v_mfma_f32_16x16x32_bf16 v[68:71], v[204:207], v[152:155], v[68:71]\n\t"
        "v_mfma_f32_16x16x32_bf16 v[100:103], v[204:207], v[156:159], v[100:103]\n\t"
        "ds_read_b128 v[204:207], %[ab] offset:41984\n\t"
        "s_waitcnt lgkmcnt(3)\n\t"
        "s_waitcnt vmcnt(7)\n\t"
        "v_mfma_f32_16x16x32_bf16 v[0:3], v[192:195], v[208:211], v[0:3]\n\t"
        "s_waitcnt vmcnt(6)\n\t"
        "v_mfma_f32_16x16x32_bf16 v[32:35], v[192:195], v[212:215], v[32:35]\n\t"
        "s_waitcnt vmcnt(5)\n\t"
        "v_mfma_f32_16x16x32_bf16 v[64:67], v[192:195], v[216:219], v[64:67]\n\t"
        "s_waitcnt vmcnt(4)\n\t"
        "v_mfma_f32_16x16x32_bf16 v[96:99], v[192:195], v[220:223], v[96:99]\n\t"
        "ds_read_b128 v[192:195], %[ab] offset:18432\n\t"
        "s_waitcnt lgkmcnt(3)\n\t"
        "v_mfma_f32_16x16x32_bf16 v[4:7], v[196:199], v[208:211], v[4:7]\n\t"
        "v_mfma_f32_16x16x32_bf16 v[36:39], v[196:199], v[212:215], v[36:39]\n\t"
        "v_mfma_f32_16x16x32_bf16 v[68:71], v[196:199], v[216:219], v[68:71]\n\t"
        "v_mfma_f32_16x16x32_bf16 v[100:103], v[196:199], v[220:223], v[100:103]\n\t"
        "ds_read_b128 v[196:199], %[ab] offset:19456\n\t"
        "s_waitcnt lgkmcnt(3)\n\t"
        "s_waitcnt vmcnt(3)\n\t"
        "v_mfma_f32_16x16x32_bf16 v[0:3], v[200:203], v[224:227], v[0:3]\n\t"
        "s_waitcnt vmcnt(2)\n\t"
        "v_mfma_f32_16x16x32_bf16 v[32:35], v[200:203], v[228:231], v[32:35]\n\t"
        "s_waitcnt vmcnt(1)\n\t"
        "v_mfma_f32_16x16x32_bf16 v[64:67], v[200:203], v[232:235], v[64:67]\n\t"
        "s_waitcnt vmcnt(0)\n\t"
        "v_mfma_f32_16x16x32_bf16 v[96:99], v[200:203], v[236:239], v[96:99]\n\t"
        "ds_read_b128 v[200:203], %[ab] offset:26624\n\t"
        "s_waitcnt lgkmcnt(3)\n\t"
        "v_mfma_f32_16x16x32_bf16 v[4:7], v[204:207], v[224:227], v[4:7]\n\t"
        "v_mfma_f32_16x16x32_bf16 v[36:39], v[204:207], v[228:231], v[36:39]\n\t"
        "v_mfma_f32_16x16x32_bf16 v[68:71], v[204:207], v[232:235], v[68:71]\n\t"
        "v_mfma_f32_16x16x32_bf16 v[100:103], v[204:207], v[236:239], v[100:103]\n\t"
        "ds_read_b128 v[204:207], %[ab] offset:27648\n\t"
        "s_waitcnt lgkmcnt(3)\n\t"
        "v_mfma_f32_16x16x32_bf16 v[8:11], v[192:195], v[128:131], v[8:11]\n\t"
        "v_mfma_f32_16x16x32_bf16 v[40:43], v[192:195], v[132:135], v[40:43]\n\t"
        "v_mfma_f32_16x16x32_bf16 v[72:75], v[192:195], v[136:139], v[72:75]\n\t"
        "v_mfma_f32_16x16x32_bf16 v[104:107], v[192:195], v[140:143], v[104:107]\n\t"
        "v_cvt_pk_bf16_f32 v0, v0, v1\n\t"
        "v_cvt_pk_bf16_f32 v1, v2, v3\n\t"
        "ds_read_b128 v[192:195], %[ab] offset:34816\n\t"
        "s_waitcnt lgkmcnt(3)\n\t"
        "v_mfma_f32_16x16x32_bf16 v[12:15], v[196:199], v[128:131], v[12:15]\n\t"
        "v_cvt_pk_bf16_f32 v2, v4, v5\n\t"
        "v_cvt_pk_bf16_f32 v3, v6, v7\n\t"
        "v_mfma_f32_16x16x32_bf16 v[44:47], v[196:199], v[132:135], v[44:47]\n\t"
        "global_store_dwordx4 %[vx], v[0:3], %[xs] offset:-4096 sc1\n\t"
        "v_cvt_pk_bf16_f32 v32, v32, v33\n\t"
        "v_cvt_pk_bf16_f32 v33, v34, v35\n\t"
        "v_mfma_f32_16x16x32_bf16 v[76:79], v[196:199], v[136:139], v[76:79]\n\t"
        "v_cvt_pk_bf16_f32 v34, v36, v37\n\t"
        "v_cvt_pk_bf16_f32 v35, v38, v39\n\t"
        "v_mfma_f32_16x16x32_bf16 v[108:111], v[196:199], v[140:143], v[108:111]\n\t"
        "global_store_dwordx4 %[vx], v[32:35], %[xs] offset:-2048 sc1\n\t"
        "v_cvt_pk_bf16_f32 v64, v64, v65\n\t"
        "v_cvt_pk_bf16_f32 v65, v66, v67\n\t"
        "ds_read_b128 v[196:199], %[ab] offset:35840\n\t"
        "s_waitcnt lgkmcnt(3)\n\t"
        "v_mfma_f32_16x16x32_bf16 v[8:11], v[200:203], v[144:147], v[8:11]\n\t"
        "s_setprio 1\n\t"
        "v_cvt_pk_bf16_f32 v66, v68, v69\n\t"
        "v_cvt_pk_bf16_f32 v67, v70, v71\n\t"
        "v_mfma_f32_16x16x32_bf16 v[40:43], v[200:203], v[148:151], v[40:43]\n\t"
        "global_store_dwordx4 %[vx], v[64:67], %[xs] offset:0 sc1\n\t"
        "v_cvt_pk_bf16_f32 v96, v96, v97\n\t"
        "v_cvt_pk_bf16_f32 v97, v98, v99\n\t"
        "v_mfma_f32_16x16x32_bf16 v[72:75], v[200:203], v[152:155], v[72:75]\n\t"
        "v_cvt_pk_bf16_f32 v98, v100, v101\n\t"
        "v_cvt_pk_bf16_f32 v99, v102, v103\n\t"
        "v_mfma_f32_16x16x32_bf16 v[104:107], v[200:203], v[156:159], v[104:107]\n\t"
        "global_store_dwordx4 %[vx], v[96:99], %[xs] offset:2048 sc1\n\t"
        "ds_read_b128 v[200:203], %[ab] offset:43008\n\t"
        "s_waitcnt lgkmcnt(3)\n\t"
        "v_mfma_f32_16x16x32_bf16 v[12:15], v[204:207], v[144:147], v[12:15]\n\t"
        "v_mfma_f32_16x16x32_bf16 v[44:47], v[204:207], v[148:151], v[44:47]\n\t"
        "v_mfma_f32_16x16x32_bf16 v[76:79], v[204:207], v[152:155], v[76:79]\n\t"
        "v_mfma_f32_16x16x32_bf16 v[108:111], v[204:207], v[156:159], v[108:111]\n\t"
        "ds_read_b128 v[204:207], %[ab] offset:44032\n\t"
        "s_waitcnt lgkmcnt(3)\n\t"
        "v_mfma_f32_16x16x32_bf16 v[8:11], v[192:195], v[208:211], v[8:11]\n\t"
        "v_mfma_f32_16x16x32_bf16 v[40:43], v[192:195], v[212:215], v[40:43]\n\t"
        "v_mfma_f32_16x16x32_bf16 v[72:75], v[192:195], v[216:219], v[72:75]\n\t"
        "v_mfma_f32_16x16x32_bf16 v[104:107], v[192:195], v[220:223], v[104:107]\n\t"
        "ds_read_b128 v[192:195], %[ab] offset:4096\n\t"
        "s_waitcnt lgkmcnt(3)\n\t"
        "v_mfma_f32_16x16x32_bf16 v[12:15], v[196:199], v[208:211], v[12:15]\n\t"
        "v_mfma_f32_16x16x32_bf16 v[44:47], v[196:199], v[212:215], v[44:47]\n\t"
        "v_mfma_f32_16x16x32_bf16 v[76:79], v[196:199], v[216:219], v[76:79]\n\t"
        "v_mfma_f32_16x16x32_bf16 v[108:111], v[196:199], v[220:223], v[108:111]\n\t"
        "ds_read_b128 v[196:199], %[ab] offset:5120\n\t"
        "s_waitcnt lgkmcnt(3)\n\t"
        "v_mfma_f32_16x16x32_bf16 v[8:11], v[200:203], v[224:227], v[8:11]\n\t"
        "v_mfma_f32_16x16x32_bf16 v[40:43], v[200:203], v[228:231], v[40:43]\n\t"
        "v_mfma_f32_16x16x32_bf16 v[72:75], v[200:203], v[232:235], v[72:75]\n\t"
        "v_mfma_f32_16x16x32_bf16 v[104:107], v[200:203], v[236:239], v[104:107]\n\t"
        "ds_read_b128 v[200:203], %[ab] offset:6144\n\t"
        "s_waitcnt lgkmcnt(3)\n\t"
        "v_mfma_f32_16x16x32_bf16 v[12:15], v[204:207], v[224:227], v[12:15]\n\t"
        "v_mfma_f32_16x16x32_bf16 v[44:47], v[204:207], v[228:231], v[44:47]\n\t"
        "v_mfma_f32_16x16x32_bf16 v[76:79], v[204:207], v[232:235], v[76:79]\n\t"
        "v_mfma_f32_16x16x32_bf16 v[108:111], v[204:207], v[236:239], v[108:111]\n\t"
        "ds_read_b128 v[204:207], %[ab] offset:7168\n\t"
        "s_waitcnt lgkmcnt(3)\n\t"
        "v_mfma_f32_16x16x32_bf16 v[16:19], v[192:195], v[160:163], v[16:19]\n\t"
        "v_mfma_f32_16x16x32_bf16 v[48:51], v[192:195], v[164:167], v[48:51]\n\t"
        "v_mfma_f32_16x16x32_bf16 v[80:83], v[192:195], v[168:171], v[80:83]\n\t"
        "v_mfma_f32_16x16x32_bf16 v[112:115], v[192:195], v[172:175], v[112:115]\n\t"
        "v_cvt_pk_bf16_f32 v8, v8, v9\n\t"
        "v_cvt_pk_bf16_f32 v9, v10, v11\n\t"
        "ds_read_b128 v[192:195], %[ab] offset:12288\n\t"
        "s_waitcnt lgkmcnt(3)\n\t"
        "v_mfma_f32_16x16x32_bf16 v[20:23], v[196:199], v[160:163], v[20:23]\n\t"
        "v_cvt_pk_bf16_f32 v10, v12, v13\n\t"
        "v_cvt_pk_bf16_f32 v11, v14, v15\n\t"
        "v_mfma_f32_16x16x32_bf16 v[52:55], v[196:199], v[164:167], v[52:55]\n\t"
        "global_store_dwordx4 %[vx], v[8:11], %[xs] offset:-3072 sc1\n\t"
        "v_cvt_pk_bf16_f32 v40, v40, v41\n\t"
        "v_cvt_pk_bf16_f32 v41, v42, v43\n\t"
        "v_mfma_f32_16x16x32_bf16 v[84:87], v[196:199], v[168:171], v[84:87]\n\t"
        "v_cvt_pk_bf16_f32 v42, v44, v45\n\t"
        "v_cvt_pk_bf16_f32 v43, v46, v47\n\t"
        "v_mfma_f32_16x16x32_bf16 v[116:119], v[196:199], v[172:175], v[116:119]\n\t"
        "global_store_dwordx4 %[vx], v[40:43], %[xs] offset:-1024 sc1\n\t"
        "v_cvt_pk_bf16_f32 v72, v72, v73\n\t"
        "v_cvt_pk_bf16_f32 v73, v74, v75\n\t"
        "ds_read_b128 v[196:199], %[ab] offset:13312\n\t"
        "s_waitcnt lgkmcnt(3)\n\t"
        "v_mfma_f32_16x16x32_bf16 v[24:27], v[200:203], v[160:163], v[24:27]\n\t"
        "v_cvt_pk_bf16_f32 v74, v76, v77\n\t"
        "v_cvt_pk_bf16_f32 v75, v78, v79\n\t"
        "v_mfma_f32_16x16x32_bf16 v[56:59], v[200:203], v[164:167], v[56:59]\n\t"
        "global_store_dwordx4 %[vx], v[72:75], %[xs] offset:1024 sc1\n\t"
        "v_cvt_pk_bf16_f32 v104, v104, v105\n\t"
        "v_cvt_pk_bf16_f32 v105, v106, v107\n\t"
        "v_mfma_f32_16x16x32_bf16 v[88:91], v[200:203], v[168:171], v[88:91]\n\t"
        "v_cvt_pk_bf16_f32 v106, v108, v109\n\t"
        "v_cvt_pk_bf16_f32 v107, v110, v111\n\t"
        "v_mfma_f32_16x16x32_bf16 v[120:123], v[200:203], v[172:175], v[120:123]\n\t"
        "global_store_dwordx4 %[vx], v[104:107], %[xs] offset:3072 sc1\n\t"
        "s_setprio 0\n\t"
        "ds_read_b128 v[200:203], %[ab] offset:14336\n\t"
        "s_waitcnt lgkmcnt(3)\n\t"
        "v_mfma_f32_16x16x32_bf16 v[28:31], v[204:207], v[160:163], v[28:31]\n\t"
        "v_mfma_f32_16x16x32_bf16 v[60:63], v[204:207], v[164:167], v[60:63]\n\t"
        "v_mfma_f32_16x16x32_bf16 v[92:95], v[204:207], v[168:171], v[92:95]\n\t"
        "v_mfma_f32_16x16x32_bf16 v[124:127], v[204:207], v[172:175], v[124:127]\n\t"
        "ds_read_b128 v[204:207], %[ab] offset:15360\n\t"
        "s_waitcnt lgkmcnt(3)\n\t"
        "v_mfma_f32_16x16x32_bf16 v[16:19], v[192:195], v[176:179], v[16:19]\n\t"
        "v_mfma_f32_16x16x32_bf16 v[48:51], v[192:195], v[180:183], v[48:51]\n\t"
        "v_mfma_f32_16x16x32_bf16 v[80:83], v[192:195], v[184:187], v[80:83]\n\t"
        "v_mfma_f32_16x16x32_bf16 v[112:115], v[192:195], v[188:191], v[112:115]\n\t"
        "ds_read_b128 v[192:195], %[ab] offset:20480\n\t"
        "s_waitcnt lgkmcnt(3)\n\t"
        "v_mfma_f32_16x16x32_bf16 v[20:23], v[196:199], v[176:179], v[20:23]\n\t"
        "v_mfma_f32_16x16x32_bf16 v[52:55], v[196:199], v[180:183], v[52:55]\n\t"
        "v_mfma_f32_16x16x32_bf16 v[84:87], v[196:199], v[184:187], v[84:87]\n\t"
        "v_mfma_f32_16x16x32_bf16 v[116:119], v[196:199], v[188:191], v[116:119]\n\t"
        "ds_read_b128 v[196:199], %[ab] offset:21504\n\t"
        "s_waitcnt lgkmcnt(3)\n\t"
        "v_mfma_f32_16x16x32_bf16 v[24:27], v[200:203], v[176:179], v[24:27]\n\t"
        "v_mfma_f32_16x16x32_bf16 v[56:59], v[200:203], v[180:183], v[56:59]\n\t"
        "v_mfma_f32_16x16x32_bf16 v[88:91], v[200:203], v[184:187], v[88:91]\n\t"
        "v_mfma_f32_16x16x32_bf16 v[120:123], v[200:203], v[188:191], v[120:123]\n\t"
        "ds_read_b128 v[200:203], %[ab] offset:22528\n\t"
        "s_waitcnt lgkmcnt(3)\n\t"
        "v_mfma_f32_16x16x32_bf16 v[28:31], v[204:207], v[176:179], v[28:31]\n\t"
        "v_mfma_f32_16x16x32_bf16 v[60:63], v[204:207], v[180:183], v[60:63]\n\t"
        "v_mfma_f32_16x16x32_bf16 v[92:95], v[204:207], v[184:187], v[92:95]\n\t"
        "v_mfma_f32_16x16x32_bf16 v[124:127], v[204:207], v[188:191], v[124:127]\n\t"
        "ds_read_b128 v[204:207], %[ab] offset:23552\n\t"
        "s_waitcnt lgkmcnt(3)\n\t"
        "v_mfma_f32_16x16x32_bf16 v[16:19], v[192:195], v[128:131], v[16:19]\n\t"
        "v_mfma_f32_16x16x32_bf16 v[48:51], v[192:195], v[132:135], v[48:51]\n\t"
        "v_mfma_f32_16x16x32_bf16 v[80:83], v[192:195], v[136:139], v[80:83]\n\t"
        "v_mfma_f32_16x16x32_bf16 v[112:115], v[192:195], v[140:143], v[112:115]\n\t"
        "ds_read_b128 v[192:195], %[ab] offset:28672\n\t"
        "s_waitcnt lgkmcnt(3)\n\t"
        "v_mfma_f32_16x16x32_bf16 v[20:23], v[196:199], v[128:131], v[20:23]\n\t"
        "v_mfma_f32_16x16x32_bf16 v[52:55], v[196:199], v[132:135], v[52:55]\n\t"
        "v_mfma_f32_16x16x32_bf16 v[84:87], v[196:199], v[136:139], v[84:87]\n\t"
        "v_mfma_f32_16x16x32_bf16 v[116:119], v[196:199], v[140:143], v[116:119]\n\t"
        "ds_read_b128 v[196:199], %[ab] offset:29696\n\t"
        "s_waitcnt lgkmcnt(3)\n\t"
        "v_mfma_f32_16x16x32_bf16 v[24:27], v[200:203], v[128:131], v[24:27]\n\t"
        "v_mfma_f32_16x16x32_bf16 v[56:59], v[200:203], v[132:135], v[56:59]\n\t"
        "v_mfma_f32_16x16x32_bf16 v[88:91], v[200:203], v[136:139], v[88:91]\n\t"
        "v_mfma_f32_16x16x32_bf16 v[120:123], v[200:203], v[140:143], v[120:123]\n\t"
        "ds_read_b128 v[200:203], %[ab] offset:30720\n\t"
        "s_waitcnt lgkmcnt(3)\n\t"
        "v_mfma_f32_16x16x32_bf16 v[28:31], v[204:207], v[128:131], v[28:31]\n\t"
        "v_mfma_f32_16x16x32_bf16 v[60:63], v[204:207], v[132:135], v[60:63]\n\t"
        "v_mfma_f32_16x16x32_bf16 v[92:95], v[204:207], v[136:139], v[92:95]\n\t"
        "v_mfma_f32_16x16x32_bf16 v[124:127], v[204:207], v[140:143], v[124:127]\n\t"
        "ds_read_b128 v[204:207], %[ab] offset:31744\n\t"
        "s_waitcnt lgkmcnt(3)\n\t"
        "v_mfma_f32_16x16x32_bf16 v[16:19], v[192:195], v[144:147], v[16:19]\n\t"
        "v_mfma_f32_16x16x32_bf16 v[48:51], v[192:195], v[148:151], v[48:51]\n\t"
        "v_mfma_f32_16x16x32_bf16 v[80:83], v[192:195], v[152:155], v[80:83]\n\t"
        "v_mfma_f32_16x16x32_bf16 v[112:115], v[192:195], v[156:159], v[112:115]\n\t"
        "ds_read_b128 v[192:195], %[ab] offset:36864\n\t"
        "s_waitcnt lgkmcnt(3)\n\t"
        "v_mfma_f32_16x16x32_bf16 v[20:23], v[196:199], v[144:147], v[20:23]\n\t"
        "v_mfma_f32_16x16x32_bf16 v[52:55], v[196:199], v[148:151], v[52:55]\n\t"
        "v_mfma_f32_16x16x32_bf16 v[84:87], v[196:199], v[152:155], v[84:87]\n\t"
        "v_mfma_f32_16x16x32_bf16 v[116:119], v[196:199], v[156:159], v[116:119]\n\t"
        "ds_read_b128 v[196:199], %[ab] offset:37888\n\t"
        "s_waitcnt lgkmcnt(3)\n\t"
        "v_mfma_f32_16x16x32_bf16 v[24:27], v[200:203], v[144:147], v[24:27]\n\t"
        "v_mfma_f32_16x16x32_bf16 v[56:59], v[200:203], v[148:151], v[56:59]\n\t"
        "v_mfma_f32_16x16x32_bf16 v[88:91], v[200:203], v[152:155], v[88:91]\n\t"
        "v_mfma_f32_16x16x32_bf16 v[120:123], v[200:203], v[156:159], v[120:123]\n\t"
        "ds_read_b128 v[200:203], %[ab] offset:38912\n\t"
        "s_waitcnt lgkmcnt(3)\n\t"
        "v_mfma_f32_16x16x32_bf16 v[28:31], v[204:207], v[144:147], v[28:31]\n\t"
        "v_mfma_f32_16x16x32_bf16 v[60:63], v[204:207], v[148:151], v[60:63]\n\t"
        "v_mfma_f32_16x16x32_bf16 v[92:95], v[204:207], v[152:155], v[92:95]\n\t"
        "v_mfma_f32_16x16x32_bf16 v[124:127], v[204:207], v[156:159], v[124:127]\n\t"
        "ds_read_b128 v[204:207], %[ab] offset:39936\n\t"
        "s_waitcnt lgkmcnt(3)\n\t"
        "v_mfma_f32_16x16x32_bf16 v[16:19], v[192:195], v[208:211], v[16:19]\n\t"
        "s_waitcnt vmcnt(0)\n\t"
        "v_mov_b32 v240, 0\n\t"
        "v_mov_b32 v241, s43\n\t"
        "s_mov_b64 exec, 1\n\t"
        "global_store_dword v240, v241, %[xf] sc1\n\t"
        "s_mov_b64 exec, -1\n\t"
        "v_mfma_f32_16x16x32_bf16 v[48:51], v[192:195], v[212:215], v[48:51]\n\t"
        "v_mfma_f32_16x16x32_bf16 v[80:83], v[192:195], v[216:219], v[80:83]\n\t"
        "v_mfma_f32_16x16x32_bf16 v[112:115], v[192:195], v[220:223], v[112:115]\n\t"
        "ds_read_b128 v[192:195], %[ab] offset:45056\n\t"
        "s_waitcnt lgkmcnt(3)\n\t"
        "v_mfma_f32_16x16x32_bf16 v[20:23], v[196:199], v[208:211], v[20:23]\n\t"
        "v_mfma_f32_16x16x32_bf16 v[52:55], v[196:199], v[212:215], v[52:55]\n\t"
        "v_mfma_f32_16x16x32_bf16 v[84:87], v[196:199], v[216:219], v[84:87]\n\t"
        "v_mfma_f32_16x16x32_bf16 v[116:119], v[196:199], v[220:223], v[116:119]\n\t"
        "ds_read_b128 v[196:199], %[ab] offset:46080\n\t"
        "s_waitcnt lgkmcnt(3)\n\t"
        "v_mfma_f32_16x16x32_bf16 v[24:27], v[200:203], v[208:211], v[24:27]\n\t"
        "v_mfma_f32_16x16x32_bf16 v[56:59], v[200:203], v[212:215], v[56:59]\n\t"
        "v_mfma_f32_16x16x32_bf16 v[88:91], v[200:203], v[216:219], v[88:91]\n\t"
        "v_mfma_f32_16x16x32_bf16 v[120:123], v[200:203], v[220:223], v[120:123]\n\t"
        "ds_read_b128 v[200:203], %[ab] offset:47104\n\t"
        "s_waitcnt lgkmcnt(3)\n\t"
        "v_mfma_f32_16x16x32_bf16 v[28:31], v[204:207], v[208:211], v[28:31]\n\t"
        "v_mfma_f32_16x16x32_bf16 v[60:63], v[204:207], v[212:215], v[60:63]\n\t"
        "v_mfma_f32_16x16x32_bf16 v[92:95], v[204:207], v[216:219], v[92:95]\n\t"
        "v_mfma_f32_16x16x32_bf16 v[124:127], v[204:207], v[220:223], v[124:127]\n\t"
        "ds_read_b128 v[204:207], %[ab] offset:48128\n\t"
        "s_waitcnt lgkmcnt(3)\n\t"
        "v_mfma_f32_16x16x32_bf16 v[16:19], v[192:195], v[224:227], v[16:19]\n\t"
        "v_mfma_f32_16x16x32_bf16 v[48:51], v[192:195], v[228:231], v[48:51]\n\t"
        "v_mfma_f32_16x16x32_bf16 v[80:83], v[192:195], v[232:235], v[80:83]\n\t"
        "v_mfma_f32_16x16x32_bf16 v[112:115], v[192:195], v[236:239], v[112:115]\n\t"
        "s_waitcnt lgkmcnt(2)\n\t"
        "v_mfma_f32_16x16x32_bf16 v[20:23], v[196:199], v[224:227], v[20:23]\n\t"
        "v_mfma_f32_16x16x32_bf16 v[52:55], v[196:199], v[228:231], v[52:55]\n\t"
        "v_mfma_f32_16x16x32_bf16 v[84:87], v[196:199], v[232:235], v[84:87]\n\t"
        "v_mfma_f32_16x16x32_bf16 v[116:119], v[196:199], v[236:239], v[116:119]\n\t"
        "s_waitcnt lgkmcnt(1)\n\t"
        "v_mfma_f32_16x16x32_bf16 v[24:27], v[200:203], v[224:227], v[24:27]\n\t"
        "v_mfma_f32_16x16x32_bf16 v[56:59], v[200:203], v[228:231], v[56:59]\n\t"
        "v_mfma_f32_16x16x32_bf16 v[88:91], v[200:203], v[232:235], v[88:91]\n\t"
        "v_mfma_f32_16x16x32_bf16 v[120:123], v[200:203], v[236:239], v[120:123]\n\t"
        "s_waitcnt lgkmcnt(0)\n\t"
        "v_mfma_f32_16x16x32_bf16 v[28:31], v[204:207], v[224:227], v[28:31]\n\t"
        "v_mfma_f32_16x16x32_bf16 v[60:63], v[204:207], v[228:231], v[60:63]\n\t"
        "v_mfma_f32_16x16x32_bf16 v[92:95], v[204:207], v[232:235], v[92:95]\n\t"
        "v_mfma_f32_16x16x32_bf16 v[124:127], v[204:207], v[236:239], v[124:127]\n\t"
        "s_branch 12f\n\t"
        "19:\n\t"
        "ds_read_b128 v[200:203], %[ab] offset:2048\n\t"
        "ds_read_b128 v[204:207], %[ab] offset:3072\n\t"
        "s_waitcnt lgkmcnt(3)\n\t"
        "v_mfma_f32_16x16x32_bf16 v[0:3], v[192:195], v[128:131], v[0:3]\n\t"
        "buffer_load_dwordx4 v[160:163], %[vx], %[srda], s46 offen offset:0 sc0 sc1\n\t"
        "s_setprio 2\n\t"
        "v_mfma_f32_16x16x32_bf16 v[32:35], v[192:195], v[132:135], v[32:35]\n\t"
        "buffer_load_dwordx4 v[164:167], %[vx], %[srda], s46 offen offset:1024 sc0 sc1\n\t"
        "v_mfma_f32_16x16x32_bf16 v[64:67], v[192:195], v[136:139], v[64:67]\n\t"
        "buffer_load_dwordx4 v[168:171], %[vx], %[srda], s47 offen offset:0 sc0 sc1\n\t"
        "v_mfma_f32_16x16x32_bf16 v[96:99], v[192:195], v[140:143], v[96:99]\n\t"
        "buffer_load_dwordx4 v[172:175], %[vx], %[srda], s47 offen offset:1024 sc0 sc1\n\t"
        "ds_read_b128 v[192:195], %[ab] offset:8192\n\t"
        "s_waitcnt lgkmcnt(3)\n\t"
        "v_mfma_f32_16x16x32_bf16 v[4:7], v[196:199], v[128:131], v[4:7]\n\t"
        "s_mov_b32 s50, 61\n\t"
        "s_bitcmp1_b64 s[44:45], s50\n\t"
        "s_cselect_b32 s46, 0x8000, 0\n\t"
        "s_lshl_b32 s47, s50, 16\n\t"
        "s_add_u32 s46, s46, s47\n\t"
        "s_xor_b32 s47, s46, 0x800\n\t"
        "buffer_load_dwordx4 v[176:179], %[vx], %[srda], s46 offen offset:0 sc0 sc1\n\t"
        "v_mfma_f32_16x16x32_bf16 v[36:39], v[196:199], v[132:135], v[36:39]\n\t"
        "buffer_load_dwordx4 v[180:183], %[vx], %[srda], s46 offen offset:1024 sc0 sc1\n\t"
        "v_mfma_f32_16x16x32_bf16 v[68:71], v[196:199], v[136:139], v[68:71]\n\t"
        "buffer_load_dwordx4 v[184:187], %[vx], %[srda], s47 offen offset:0 sc0 sc1\n\t"
        "v_mfma_f32_16x16x32_bf16 v[100:103], v[196:199], v[140:143], v[100:103]\n\t"
        "buffer_load_dwordx4 v[188:191], %[vx], %[srda], s47 offen offset:1024 sc0 sc1\n\t"
        "ds_read_b128 v[196:199], %[ab] offset:9216\n\t"
        "s_waitcnt lgkmcnt(3)\n\t"
        "v_mfma_f32_16x16x32_bf16 v[8:11], v[200:203], v[128:131], v[8:11]\n\t"
        "s_mov_b32 s50, 62\n\t"
        "s_bitcmp1_b64 s[44:45], s50\n\t"
        "s_cselect_b32 s46, 0x8000, 0\n\t"
        "s_lshl_b32 s47, s50, 16\n\t"
        "s_add_u32 s46, s46, s47\n\t"
        "s_xor_b32 s47, s46, 0x800\n\t"
        "buffer_load_dwordx4 v[208:211], %[vx], %[srda], s46 offen offset:0 sc0 sc1\n\t"
        "v_mfma_f32_16x16x32_bf16 v[40:43], v[200:203], v[132:135], v[40:43]\n\t"
        "buffer_load_dwordx4 v[212:215], %[vx], %[srda], s46 offen offset:1024 sc0 sc1\n\t"
        "v_mfma_f32_16x16x32_bf16 v[72:75], v[200:203], v[136:139], v[72:75]\n\t"
        "buffer_load_dwordx4 v[216:219], %[vx], %[srda], s47 offen offset:0 sc0 sc1\n\t"
        "v_mfma_f32_16x16x32_bf16 v[104:107], v[200:203], v[140:143], v[104:107]\n\t"
        "buffer_load_dwordx4 v[220:223], %[vx], %[srda], s47 offen offset:1024 sc0 sc1\n\t"
        "ds_read_b128 v[200:203], %[ab] offset:10240\n\t"
        "s_waitcnt lgkmcnt(3)\n\t"
        "v_mfma_f32_16x16x32_bf16 v[12:15], v[204:207], v[128:131], v[12:15]\n\t"
        "s_mov_b32 s50, 63\n\t"
        "s_bitcmp1_b64 s[44:45], s50\n\t"
        "s_cselect_b32 s46, 0x8000, 0\n\t"
        "s_lshl_b32 s47, s50, 16\n\t"
        "s_add_u32 s46, s46, s47\n\t"
        "s_xor_b32 s47, s46, 0x800\n\t"
        "buffer_load_dwordx4 v[224:227], %[vx], %[srda], s46 offen offset:0 sc0 sc1\n\t"
        "v_mfma_f32_16x16x32_bf16 v[44:47], v[204:207], v[132:135], v[44:47]\n\t"
        "buffer_load_dwordx4 v[228:231], %[vx], %[srda], s46 offen offset:1024 sc0 sc1\n\t"
        "v_mfma_f32_16x16x32_bf16 v[76:79], v[204:207], v[136:139], v[76:79]\n\t"
        "buffer_load_dwordx4 v[232:235], %[vx], %[srda], s47 offen offset:0 sc0 sc1\n\t"
        "v_mfma_f32_16x16x32_bf16 v[108:111], v[204:207], v[140:143], v[108:111]\n\t"
        "buffer_load_dwordx4 v[236:239], %[vx], %[srda], s47 offen offset:1024 sc0 sc1\n\t"
        "ds_read_b128 v[204:207], %[ab] offset:11264\n\t"
        "s_waitcnt lgkmcnt(3)\n\t"
        "v_mfma_f32_16x16x32_bf16 v[0:3], v[192:195], v[144:147], v[0:3]\n\t"
        "v_mfma_f32_16x16x32_bf16 v[32:35], v[192:195], v[148:151], v[32:35]\n\t"
        "v_mfma_f32_16x16x32_bf16 v[64:67], v[192:195], v[152:155], v[64:67]\n\t"
        "v_mfma_f32_16x16x32_bf16 v[96:99], v[192:195], v[156:159], v[96:99]\n\t"
        "ds_read_b128 v[192:195], %[ab] offset:16384\n\t"
        "s_waitcnt lgkmcnt(3)\n\t"
        "v_mfma_f32_16x16x32_bf16 v[4:7], v[196:199], v[144:147], v[4:7]\n\t"
        "v_mfma_f32_16x16x32_bf16 v[36:39], v[196:199], v[148:151], v[36:39]\n\t"
        "v_mfma_f32_16x16x32_bf16 v[68:71], v[196:199], v[152:155], v[68:71]\n\t"
        "v_mfma_f32_16x16x32_bf16 v[100:103], v[196:199], v[156:159], v[100:103]\n\t"
        "ds_read_b128 v[196:199], %[ab] offset:17408\n\t"
        "s_waitcnt lgkmcnt(3)\n\t"
        "v_mfma_f32_16x16x32_bf16 v[8:11], v[200:203], v[144:147], v[8:11]\n\t"
        "v_mfma_f32_16x16x32_bf16 v[40:43], v[200:203], v[148:151], v[40:43]\n\t"
        "v_mfma_f32_16x16x32_bf16 v[72:75], v[200:203], v[152:155], v[72:75]\n\t"
        "v_mfma_f32_16x16x32_bf16 v[104:107], v[200:203], v[156:159], v[104:107]\n\t"
        "ds_read_b128 v[200:203], %[ab] offset:24576\n\t"
        "s_waitcnt lgkmcnt(3)\n\t"
        "v_mfma_f32_16x16x32_bf16 v[12:15], v[204:207], v[144:147], v[12:15]\n\t"
        "v_mfma_f32_16x16x32_bf16 v[44:47], v[204:207], v[148:151], v[44:47]\n\t"
        "v_mfma_f32_16x16x32_bf16 v[76:79], v[204:207], v[152:155], v[76:79]\n\t"
        "v_mfma_f32_16x16x32_bf16 v[108:111], v[204:207], v[156:159], v[108:111]\n\t"
        "ds_read_b128 v[204:207], %[ab] offset:25600\n\t"
        "s_waitcnt lgkmcnt(3)\n\t"
        "s_waitcnt vmcnt(15)\n\t"
        "v_mfma_f32_16x16x32_bf16 v[0:3], v[192:195], v[160:163], v[0:3]\n\t"
        "s_waitcnt vmcnt(14)\n\t"
        "v_mfma_f32_16x16x32_bf16 v[32:35], v[192:195], v[164:167], v[32:35]\n\t"
        "s_waitcnt vmcnt(13)\n\t"
        "v_mfma_f32_16x16x32_bf16 v[64:67], v[192:195], v[168:171], v[64:67]\n\t"
        "s_waitcnt vmcnt(12)\n\t"
        "v_mfma_f32_16x16x32_bf16 v[96:99], v[192:195], v[172:175], v[96:99]\n\t"
        "ds_read_b128 v[192:195], %[ab] offset:32768\n\t"
        "s_waitcnt lgkmcnt(3)\n\t"
        "v_mfma_f32_16x16x32_bf16 v[4:7], v[196:199], v[160:163], v[4:7]\n\t"
        "v_mfma_f32_16x16x32_bf16 v[36:39], v[196:199], v[164:167], v[36:39]\n\t"
        "v_mfma_f32_16x16x32_bf16 v[68:71], v[196:199], v[168:171], v[68:71]\n\t"
        "v_mfma_f32_16x16x32_bf16 v[100:103], v[196:199], v[172:175], v[100:103]\n\t"
        "ds_read_b128 v[196:199], %[ab] offset:33792\n\t"
        "s_waitcnt lgkmcnt(3)\n\t"
        "s_waitcnt vmcnt(11)\n\t"
        "v_mfma_f32_16x16x32_bf16 v[0:3], v[200:203], v[176:179], v[0:3]\n\t"
        "s_waitcnt vmcnt(10)\n\t"
        "v_mfma_f32_16x16x32_bf16 v[32:35], v[200:203], v[180:183], v[32:35]\n\t"
        "s_waitcnt vmcnt(9)\n\t"
        "v_mfma_f32_16x16x32_bf16 v[64:67], v[200:203], v[184:187], v[64:67]\n\t"
        "s_waitcnt vmcnt(8)\n\t"
        "v_mfma_f32_16x16x32_bf16 v[96:99], v[200:203], v[188:191], v[96:99]\n\t"
        "ds_read_b128 v[200:203], %[ab] offset:40960\n\t"
        "s_waitcnt lgkmcnt(3)\n\t"
        "v_mfma_f32_16x16x32_bf16 v[4:7], v[204:207], v[176:179], v[4:7]\n\t"
        "v_mfma_f32_16x16x32_bf16 v[36:39], v[204:207], v[180:183], v[36:39]\n\t"
        "v_mfma_f32_16x16x32_bf16 v[68:71], v[204:207], v[184:187], v[68:71]\n\t"
        "v_mfma_f32_16x16x32_bf16 v[100:103], v[204:207], v[188:191], v[100:103]\n\t"
        "ds_read_b128 v[204:207], %[ab] offset:41984\n\t"
        "s_waitcnt lgkmcnt(3)\n\t"
        "s_waitcnt vmcnt(7)\n\t"
        "v_mfma_f32_16x16x32_bf16 v[0:3], v[192:195], v[208:211], v[0:3]\n\t"
        "s_waitcnt vmcnt(6)\n\t"
        "v_mfma_f32_16x16x32_bf16 v[32:35], v[192:195], v[212:215], v[32:35]\n\t"
        "s_waitcnt vmcnt(5)\n\t"
        "v_mfma_f32_16x16x32_bf16 v[64:67], v[192:195], v[216:219], v[64:67]\n\t"
        "s_waitcnt vmcnt(4)\n\t"
        "v_mfma_f32_16x16x32_bf16 v[96:99], v[192:195], v[220:223], v[96:99]\n\t"
        "ds_read_b128 v[192:195], %[ab] offset:18432\n\t"
        "s_waitcnt lgkmcnt(3)\n\t"
        "v_mfma_f32_16x16x32_bf16 v[4:7], v[196:199], v[208:211], v[4:7]\n\t"
        "v_mfma_f32_16x16x32_bf16 v[36:39], v[196:199], v[212:215], v[36:39]\n\t"
        "v_mfma_f32_16x16x32_bf16 v[68:71], v[196:199], v[216:219], v[68:71]\n\t"
        "v_mfma_f32_16x16x32_bf16 v[100:103], v[196:199], v[220:223], v[100:103]\n\t"
        "ds_read_b128 v[196:199], %[ab] offset:19456\n\t"
        "s_waitcnt lgkmcnt(3)\n\t"
        "s_waitcnt vmcnt(3)\n\t"
        "v_mfma_f32_16x16x32_bf16 v[0:3], v[200:203], v[224:227], v[0:3]\n\t"
        "s_waitcnt vmcnt(2)\n\t"
        "v_mfma_f32_16x16x32_bf16 v[32:35], v[200:203], v[228:231], v[32:35]\n\t"
        "s_waitcnt vmcnt(1)\n\t"
        "v_mfma_f32_16x16x32_bf16 v[64:67], v[200:203], v[232:235], v[64:67]\n\t"
        "s_waitcnt vmcnt(0)\n\t"
        "v_mfma_f32_16x16x32_bf16 v[96:99], v[200:203], v[236:239], v[96:99]\n\t"
        "ds_read_b128 v[200:203], %[ab] offset:26624\n\t"
        "s_waitcnt lgkmcnt(3)\n\t"
        "v_mfma_f32_16x16x32_bf16 v[4:7], v[204:207], v[224:227], v[4:7]\n\t"
        "v_mfma_f32_16x16x32_bf16 v[36:39], v[204:207], v[228:231], v[36:39]\n\t"
        "v_mfma_f32_16x16x32_bf16 v[68:71], v[204:207], v[232:235], v[68:71]\n\t"
        "v_mfma_f32_16x16x32_bf16 v[100:103], v[204:207], v[236:239], v[100:103]\n\t"
        "ds_read_b128 v[204:207], %[ab] offset:27648\n\t"
        "s_waitcnt lgkmcnt(3)\n\t"
        "v_mfma_f32_16x16x32_bf16 v[8:11], v[192:195], v[160:163], v[8:11]\n\t"
        "v_mfma_f32_16x16x32_bf16 v[40:43], v[192:195], v[164:167], v[40:43]\n\t"
        "v_mfma_f32_16x16x32_bf16 v[72:75], v[192:195], v[168:171], v[72:75]\n\t"
        "v_mfma_f32_16x16x32_bf16 v[104:107], v[192:195], v[172:175], v[104:107]\n\t"
        "v_cvt_pk_bf16_f32 v0, v0, v1\n\t"
        "v_cvt_pk_bf16_f32 v1, v2, v3\n\t"
        "ds_read_b128 v[192:195], %[ab] offset:34816\n\t"
        "s_waitcnt lgkmcnt(3)\n\t"
        "v_mfma_f32_16x16x32_bf16 v[12:15], v[196:199], v[160:163], v[12:15]\n\t"
        "v_cvt_pk_bf16_f32 v2, v4, v5\n\t"
        "v_cvt_pk_bf16_f32 v3, v6, v7\n\t"
        "v_mfma_f32_16x16x32_bf16 v[44:47], v[196:199], v[164:167], v[44:47]\n\t"
        "global_store_dwordx4 %[vx], v[0:3], %[xs] offset:-4096 sc1\n\t"
        "v_cvt_pk_bf16_f32 v32, v32, v33\n\t"
        "v_cvt_pk_bf16_f32 v33, v34, v35\n\t"
        "v_mfma_f32_16x16x32_bf16 v[76:79], v[196:199], v[168:171], v[76:79]\n\t"
        "v_cvt_pk_bf16_f32 v34, v36, v37\n\t"
        "v_cvt_pk_bf16_f32 v35, v38, v39\n\t"
        "v_mfma_f32_16x16x32_bf16 v[108:111], v[196:199], v[172:175], v[108:111]\n\t"
        "global_store_dwordx4 %[vx], v[32:35], %[xs] offset:-2048 sc1\n\t"
        "v_cvt_pk_bf16_f32 v64, v64, v65\n\t"
        "v_cvt_pk_bf16_f32 v65, v66, v67\n\t"
        "ds_read_b128 v[196:199], %[ab] offset:35840\n\t"
        "s_waitcnt lgkmcnt(3)\n\t"
        "v_mfma_f32_16x16x32_bf16 v[8:11], v[200:203], v[176:179], v[8:11]\n\t"
        "s_setprio 1\n\t"
        "v_cvt_pk_bf16_f32 v66, v68, v69\n\t"
        "v_cvt_pk_bf16_f32 v67, v70, v71\n\t"
        "v_mfma_f32_16x16x32_bf16 v[40:43], v[200:203], v[180:183], v[40:43]\n\t"
        "global_store_dwordx4 %[vx], v[64:67], %[xs] offset:0 sc1\n\t"
        "v_cvt_pk_bf16_f32 v96, v96, v97\n\t"
        "v_cvt_pk_bf16_f32 v97, v98, v99\n\t"
        "v_mfma_f32_16x16x32_bf16 v[72:75], v[200:203], v[184:187], v[72:75]\n\t"
        "v_cvt_pk_bf16_f32 v98, v100, v101\n\t"
        "v_cvt_pk_bf16_f32 v99, v102, v103\n\t"
        "v_mfma_f32_16x16x32_bf16 v[104:107], v[200:203], v[188:191], v[104:107]\n\t"
        "global_store_dwordx4 %[vx], v[96:99], %[xs] offset:2048 sc1\n\t"
        "ds_read_b128 v[200:203], %[ab] offset:43008\n\t"
        "s_waitcnt lgkmcnt(3)\n\t"
        "v_mfma_f32_16x16x32_bf16 v[12:15], v[204:207], v[176:179], v[12:15]\n\t"
        "v_mfma_f32_16x16x32_bf16 v[44:47], v[204:207], v[180:183], v[44:47]\n\t"
        "v_mfma_f32_16x16x32_bf16 v[76:79], v[204:207], v[184:187], v[76:79]\n\t"
        "v_mfma_f32_16x16x32_bf16 v[108:111], v[204:207], v[188:191], v[108:111]\n\t"
        "ds_read_b128 v[204:207], %[ab] offset:44032\n\t"
        "s_waitcnt lgkmcnt(3)\n\t"
        "v_mfma_f32_16x16x32_bf16 v[8:11], v[192:195], v[208:211], v[8:11]\n\t"
        "v_mfma_f32_16x16x32_bf16 v[40:43], v[192:195], v[212:215], v[40:43]\n\t"
        "v_mfma_f32_16x16x32_bf16 v[72:75], v[192:195], v[216:219], v[72:75]\n\t"
        "v_mfma_f32_16x16x32_bf16 v[104:107], v[192:195], v[220:223], v[104:107]\n\t"
        "ds_read_b128 v[192:195], %[ab] offset:4096\n\t"
        "s_waitcnt lgkmcnt(3)\n\t"
        "v_mfma_f32_16x16x32_bf16 v[12:15], v[196:199], v[208:211], v[12:15]\n\t"
        "v_mfma_f32_16x16x32_bf16 v[44:47], v[196:199], v[212:215], v[44:47]\n\t"
        "v_mfma_f32_16x16x32_bf16 v[76:79], v[196:199], v[216:219], v[76:79]\n\t"
        "v_mfma_f32_16x16x32_bf16 v[108:111], v[196:199], v[220:223], v[108:111]\n\t"
        "ds_read_b128 v[196:199], %[ab] offset:5120\n\t"
        "s_waitcnt lgkmcnt(3)\n\t"
        "v_mfma_f32_16x16x32_bf16 v[8:11], v[200:203], v[224:227], v[8:11]\n\t"
        "v_mfma_f32_16x16x32_bf16 v[40:43], v[200:203], v[228:231], v[40:43]\n\t"
        "v_mfma_f32_16x16x32_bf16 v[72:75], v[200:203], v[232:235], v[72:75]\n\t"
        "v_mfma_f32_16x16x32_bf16 v[104:107], v[200:203], v[236:239], v[104:107]\n\t"
        "ds_read_b128 v[200:203], %[ab] offset:6144\n\t"
        "s_waitcnt lgkmcnt(3)\n\t"
        "v_mfma_f32_16x16x32_bf16 v[12:15], v[204:207], v[224:227], v[12:15]\n\t"
        "v_mfma_f32_16x16x32_bf16 v[44:47], v[204:207], v[228:231], v[44:47]\n\t"
        "v_mfma_f32_16x16x32_bf16 v[76:79], v[204:207], v[232:235], v[76:79]\n\t"
        "v_mfma_f32_16x16x32_bf16 v[108:111], v[204:207], v[236:239], v[108:111]\n\t"
        "ds_read_b128 v[204:207], %[ab] offset:7168\n\t"
        "s_waitcnt lgkmcnt(3)\n\t"
        "v_mfma_f32_16x16x32_bf16 v[16:19], v[192:195], v[128:131], v[16:19]\n\t"
        "v_mfma_f32_16x16x32_bf16 v[48:51], v[192:195], v[132:135], v[48:51]\n\t"
        "v_mfma_f32_16x16x32_bf16 v[80:83], v[192:195], v[136:139], v[80:83]\n\t"
        "v_mfma_f32_16x16x32_bf16 v[112:115], v[192:195], v[140:143], v[112:115]\n\t"
        "v_cvt_pk_bf16_f32 v8, v8, v9\n\t"
        "v_cvt_pk_bf16_f32 v9, v10, v11\n\t"
        "ds_read_b128 v[192:195], %[ab] offset:12288\n\t"
        "s_waitcnt lgkmcnt(3)\n\t"
        "v_mfma_f32_16x16x32_bf16 v[20:23], v[196:199], v[128:131], v[20:23]\n\t"
        "v_cvt_pk_bf16_f32 v10, v12, v13\n\t"
        "v_cvt_pk_bf16_f32 v11, v14, v15\n\t"
        "v_mfma_f32_16x16x32_bf16 v[52:55], v[196:199], v[132:135], v[52:55]\n\t"
        "global_store_dwordx4 %[vx], v[8:11], %[xs] offset:-3072 sc1\n\t"
        "v_cvt_pk_bf16_f32 v40, v40, v41\n\t"
        "v_cvt_pk_bf16_f32 v41, v42, v43\n\t"
        "v_mfma_f32_16x16x32_bf16 v[84:87], v[196:199], v[136:139], v[84:87]\n\t"
        "v_cvt_pk_bf16_f32 v42, v44, v45\n\t"
        "v_cvt_pk_bf16_f32 v43, v46, v47\n\t"
        "v_mfma_f32_16x16x32_bf16 v[116:119], v[196:199], v[140:143], v[116:119]\n\t"
        "global_store_dwordx4 %[vx], v[40:43], %[xs] offset:-1024 sc1\n\t"
        "v_cvt_pk_bf16_f32 v72, v72, v73\n\t"
        "v_cvt_pk_bf16_f32 v73, v74, v75\n\t"
        "ds_read_b128 v[196:199], %[ab] offset:13312\n\t"
        "s_waitcnt lgkmcnt(3)\n\t"
        "v_mfma_f32_16x16x32_bf16 v[24:27], v[200:203], v[128:131], v[24:27]\n\t"
        "v_cvt_pk_bf16_f32 v74, v76, v77\n\t"
        "v_cvt_pk_bf16_f32 v75, v78, v79\n\t"
        "v_mfma_f32_16x16x32_bf16 v[56:59], v[200:203], v[132:135], v[56:59]\n\t"
        "global_store_dwordx4 %[vx], v[72:75], %[xs] offset:1024 sc1\n\t"
        "v_cvt_pk_bf16_f32 v104, v104, v105\n\t"
        "v_cvt_pk_bf16_f32 v105, v106, v107\n\t"
        "v_mfma_f32_16x16x32_bf16 v[88:91], v[200:203], v[136:139], v[88:91]\n\t"
        "v_cvt_pk_bf16_f32 v106, v108, v109\n\t"
        "v_cvt_pk_bf16_f32 v107, v110, v111\n\t"
        "v_mfma_f32_16x16x32_bf16 v[120:123], v[200:203], v[140:143], v[120:123]\n\t"
        "global_store_dwordx4 %[vx], v[104:107], %[xs] offset:3072 sc1\n\t"
        "s_setprio 0\n\t"
        "ds_read_b128 v[200:203], %[ab] offset:14336\n\t"
        "s_waitcnt lgkmcnt(3)\n\t"
        "v_mfma_f32_16x16x32_bf16 v[28:31], v[204:207], v[128:131], v[28:31]\n\t"
        "v_mfma_f32_16x16x32_bf16 v[60:63], v[204:207], v[132:135], v[60:63]\n\t"
        "v_mfma_f32_16x16x32_bf16 v[92:95], v[204:207], v[136:139], v[92:95]\n\t"
        "v_mfma_f32_16x16x32_bf16 v[124:127], v[204:207], v[140:143], v[124:127]\n\t"
        "ds_read_b128 v[204:207], %[ab] offset:15360\n\t"
        "s_waitcnt lgkmcnt(3)\n\t"
        "v_mfma_f32_16x16x32_bf16 v[16:19], v[192:195], v[144:147], v[16:19]\n\t"
        "v_mfma_f32_16x16x32_bf16 v[48:51], v[192:195], v[148:151], v[48:51]\n\t"
        "v_mfma_f32_16x16x32_bf16 v[80:83], v[192:195], v[152:155], v[80:83]\n\t"
        "v_mfma_f32_16x16x32_bf16 v[112:115], v[192:195], v[156:159], v[112:115]\n\t"
        "ds_read_b128 v[192:195], %[ab] offset:20480\n\t"
        "s_waitcnt lgkmcnt(3)\n\t"
        "v_mfma_f32_16x16x32_bf16 v[20:23], v[196:199], v[144:147], v[20:23]\n\t"
        "v_mfma_f32_16x16x32_bf16 v[52:55], v[196:199], v[148:151], v[52:55]\n\t"
        "v_mfma_f32_16x16x32_bf16 v[84:87], v[196:199], v[152:155], v[84:87]\n\t"
        "v_mfma_f32_16x16x32_bf16 v[116:119], v[196:199], v[156:159], v[116:119]\n\t"
        "ds_read_b128 v[196:199], %[ab] offset:21504\n\t"
        "s_waitcnt lgkmcnt(3)\n\t"
        "v_mfma_f32_16x16x32_bf16 v[24:27], v[200:203], v[144:147], v[24:27]\n\t"
        "v_mfma_f32_16x16x32_bf16 v[56:59], v[200:203], v[148:151], v[56:59]\n\t"
        "v_mfma_f32_16x16x32_bf16 v[88:91], v[200:203], v[152:155], v[88:91]\n\t"
        "v_mfma_f32_16x16x32_bf16 v[120:123], v[200:203], v[156:159], v[120:123]\n\t"
        "ds_read_b128 v[200:203], %[ab] offset:22528\n\t"
        "s_waitcnt lgkmcnt(3)\n\t"
        "v_mfma_f32_16x16x32_bf16 v[28:31], v[204:207], v[144:147], v[28:31]\n\t"
        "v_mfma_f32_16x16x32_bf16 v[60:63], v[204:207], v[148:151], v[60:63]\n\t"
        "v_mfma_f32_16x16x32_bf16 v[92:95], v[204:207], v[152:155], v[92:95]\n\t"
        "v_mfma_f32_16x16x32_bf16 v[124:127], v[204:207], v[156:159], v[124:127]\n\t"
        "ds_read_b128 v[204:207], %[ab] offset:23552\n\t"
        "s_waitcnt lgkmcnt(3)\n\t"
        "v_mfma_f32_16x16x32_bf16 v[16:19], v[192:195], v[160:163], v[16:19]\n\t"
        "v_mfma_f32_16x16x32_bf16 v[48:51], v[192:195], v[164:167], v[48:51]\n\t"
        "v_mfma_f32_16x16x32_bf16 v[80:83], v[192:195], v[168:171], v[80:83]\n\t"
        "v_mfma_f32_16x16x32_bf16 v[112:115], v[192:195], v[172:175], v[112:115]\n\t"
        "ds_read_b128 v[192:195], %[ab] offset:28672\n\t"
        "s_waitcnt lgkmcnt(3)\n\t"
        "v_mfma_f32_16x16x32_bf16 v[20:23], v[196:199], v[160:163], v[20:23]\n\t"
        "v_mfma_f32_16x16x32_bf16 v[52:55], v[196:199], v[164:167], v[52:55]\n\t"
        "v_mfma_f32_16x16x32_bf16 v[84:87], v[196:199], v[168:171], v[84:87]\n\t"
        "v_mfma_f32_16x16x32_bf16 v[116:119], v[196:199], v[172:175], v[116:119]\n\t"
        "ds_read_b128 v[196:199], %[ab] offset:29696\n\t"
        "s_waitcnt lgkmcnt(3)\n\t"
        "v_mfma_f32_16x16x32_bf16 v[24:27], v[200:203], v[160:163], v[24:27]\n\t"
        "v_mfma_f32_16x16x32_bf16 v[56:59], v[200:203], v[164:167], v[56:59]\n\t"
        "v_mfma_f32_16x16x32_bf16 v[88:91], v[200:203], v[168:171], v[88:91]\n\t"
        "v_mfma_f32_16x16x32_bf16 v[120:123], v[200:203], v[172:175], v[120:123]\n\t"
        "ds_read_b128 v[200:203], %[ab] offset:30720\n\t"
        "s_waitcnt lgkmcnt(3)\n\t"
        "v_mfma_f32_16x16x32_bf16 v[28:31], v[204:207], v[160:163], v[28:31]\n\t"
        "v_mfma_f32_16x16x32_bf16 v[60:63], v[204:207], v[164:167], v[60:63]\n\t"
        "v_mfma_f32_16x16x32_bf16 v[92:95], v[204:207], v[168:171], v[92:95]\n\t"
        "v_mfma_f32_16x16x32_bf16 v[124:127], v[204:207], v[172:175], v[124:127]\n\t"
        "ds_read_b128 v[204:207], %[ab] offset:31744\n\t"
        "s_waitcnt lgkmcnt(3)\n\t"
        "v_mfma_f32_16x16x32_bf16 v[16:19], v[192:195], v[176:179], v[16:19]\n\t"
        "v_mfma_f32_16x16x32_bf16 v[48:51], v[192:195], v[180:183], v[48:51]\n\t"
        "v_mfma_f32_16x16x32_bf16 v[80:83], v[192:195], v[184:187], v[80:83]\n\t"
        "v_mfma_f32_16x16x32_bf16 v[112:115], v[192:195], v[188:191], v[112:115]\n\t"
        "ds_read_b128 v[192:195], %[ab] offset:36864\n\t"
        "s_waitcnt lgkmcnt(3)\n\t"
        "v_mfma_f32_16x16x32_bf16 v[20:23], v[196:199], v[176:179], v[20:23]\n\t"
        "v_mfma_f32_16x16x32_bf16 v[52:55], v[196:199], v[180:183], v[52:55]\n\t"
        "v_mfma_f32_16x16x32_bf16 v[84:87], v[196:199], v[184:187], v[84:87]\n\t"
        "v_mfma_f32_16x16x32_bf16 v[116:119], v[196:199], v[188:191], v[116:119]\n\t"
        "ds_read_b128 v[196:199], %[ab] offset:37888\n\t"
        "s_waitcnt lgkmcnt(3)\n\t"
        "v_mfma_f32_16x16x32_bf16 v[24:27], v[200:203], v[176:179], v[24:27]\n\t"
        "v_mfma_f32_16x16x32_bf16 v[56:59], v[200:203], v[180:183], v[56:59]\n\t"
        "v_mfma_f32_16x16x32_bf16 v[88:91], v[200:203], v[184:187], v[88:91]\n\t"
        "v_mfma_f32_16x16x32_bf16 v[120:123], v[200:203], v[188:191], v[120:123]\n\t"
        "ds_read_b128 v[200:203], %[ab] offset:38912\n\t"
        "s_waitcnt lgkmcnt(3)\n\t"
        "v_mfma_f32_16x16x32_bf16 v[28:31], v[204:207], v[176:179], v[28:31]\n\t"
        "v_mfma_f32_16x16x32_bf16 v[60:63], v[204:207], v[180:183], v[60:63]\n\t"
        "v_mfma_f32_16x16x32_bf16 v[92:95], v[204:207], v[184:187], v[92:95]\n\t"
        "v_mfma_f32_16x16x32_bf16 v[124:127], v[204:207], v[188:191], v[124:127]\n\t"
        "ds_read_b128 v[204:207], %[ab] offset:39936\n\t"
        "s_waitcnt lgkmcnt(3)\n\t"
        "v_mfma_f32_16x16x32_bf16 v[16:19], v[192:195], v[208:211], v[16:19]\n\t"
        "s_waitcnt vmcnt(0)\n\t"
        "v_mov_b32 v240, 0\n\t"
        "v_mov_b32 v241, s43\n\t"
        "s_mov_b64 exec, 1\n\t"
        "global_store_dword v240, v241, %[xf] sc1\n\t"
        "s_mov_b64 exec, -1\n\t"
        "v_mfma_f32_16x16x32_bf16 v[48:51], v[192:195], v[212:215], v[48:51]\n\t"
        "v_mfma_f32_16x16x32_bf16 v[80:83], v[192:195], v[216:219], v[80:83]\n\t"
        "v_mfma_f32_16x16x32_bf16 v[112:115], v[192:195], v[220:223], v[112:115]\n\t"
        "ds_read_b128 v[192:195], %[ab] offset:45056\n\t"
        "s_waitcnt lgkmcnt(3)\n\t"
        "v_mfma_f32_16x16x32_bf16 v[20:23], v[196:199], v[208:211], v[20:23]\n\t"
        "v_mfma_f32_16x16x32_bf16 v[52:55], v[196:199], v[212:215], v[52:55]\n\t"
        "v_mfma_f32_16x16x32_bf16 v[84:87], v[196:199], v[216:219], v[84:87]\n\t"
        "v_mfma_f32_16x16x32_bf16 v[116:119], v[196:199], v[220:223], v[116:119]\n\t"
        "ds_read_b128 v[196:199], %[ab] offset:46080\n\t"
        "s_waitcnt lgkmcnt(3)\n\t"
        "v_mfma_f32_16x16x32_bf16 v[24:27], v[200:203], v[208:211], v[24:27]\n\t"
        "v_mfma_f32_16x16x32_bf16 v[56:59], v[200:203], v[212:215], v[56:59]\n\t"
        "v_mfma_f32_16x16x32_bf16 v[88:91], v[200:203], v[216:219], v[88:91]\n\t"
        "v_mfma_f32_16x16x32_bf16 v[120:123], v[200:203], v[220:223], v[120:123]\n\t"
        "ds_read_b128 v[200:203], %[ab] offset:47104\n\t"
        "s_waitcnt lgkmcnt(3)\n\t"
        "v_mfma_f32_16x16x32_bf16 v[28:31], v[204:207], v[208:211], v[28:31]\n\t"
        "v_mfma_f32_16x16x32_bf16 v[60:63], v[204:207], v[212:215], v[60:63]\n\t"
        "v_mfma_f32_16x16x32_bf16 v[92:95], v[204:207], v[216:219], v[92:95]\n\t"
        "v_mfma_f32_16x16x32_bf16 v[124:127], v[204:207], v[220:223], v[124:127]\n\t"
        "ds_read_b128 v[204:207], %[ab] offset:48128\n\t"
        "s_waitcnt lgkmcnt(3)\n\t"
        "v_mfma_f32_16x16x32_bf16 v[16:19], v[192:195], v[224:227], v[16:19]\n\t"
        "v_mfma_f32_16x16x32_bf16 v[48:51], v[192:195], v[228:231], v[48:51]\n\t"
        "v_mfma_f32_16x16x32_bf16 v[80:83], v[192:195], v[232:235], v[80:83]\n\t"
        "v_mfma_f32_16x16x32_bf16 v[112:115], v[192:195], v[236:239], v[112:115]\n\t"
        "s_waitcnt lgkmcnt(2)\n\t"
        "v_mfma_f32_16x16x32_bf16 v[20:23], v[196:199], v[224:227], v[20:23]\n\t"
        "v_mfma_f32_16x16x32_bf16 v[52:55], v[196:199], v[228:231], v[52:55]\n\t"
        "v_mfma_f32_16x16x32_bf16 v[84:87], v[196:199], v[232:235], v[84:87]\n\t"
        "v_mfma_f32_16x16x32_bf16 v[116:119], v[196:199], v[236:239], v[116:119]\n\t"
        "s_waitcnt lgkmcnt(1)\n\t"
        "v_mfma_f32_16x16x32_bf16 v[24:27], v[200:203], v[224:227], v[24:27]\n\t"
        "v_mfma_f32_16x16x32_bf16 v[56:59], v[200:203], v[228:231], v[56:59]\n\t"
        "v_mfma_f32_16x16x32_bf16 v[88:91], v[200:203], v[232:235], v[88:91]\n\t"
        "v_mfma_f32_16x16x32_bf16 v[120:123], v[200:203], v[236:239], v[120:123]\n\t"
        "s_waitcnt lgkmcnt(0)\n\t"
        "v_mfma_f32_16x16x32_bf16 v[28:31], v[204:207], v[224:227], v[28:31]\n\t"
        "v_mfma_f32_16x16x32_bf16 v[60:63], v[204:207], v[228:231], v[60:63]\n\t"
        "v_mfma_f32_16x16x32_bf16 v[92:95], v[204:207], v[232:235], v[92:95]\n\t"
        "v_mfma_f32_16x16x32_bf16 v[124:127], v[204:207], v[236:239], v[124:127]\n\t"
        "12:\n\t"
        "s_waitcnt lgkmcnt(0)\n\t"
        "s_mov_b32 %[oc], s43\n\t"
        "s_setprio 0\n\t"
        "s_nop 15\n\t"
        "s_nop 15\n\t"
            : "=&{v[0:15]}"(av[0]), "=&{v[16:31]}"(av[1]), "=&{v[32:47]}"(av[2]), "=&{v[48:63]}"(av[3]),
              "=&{v[64:79]}"(av[4]), "=&{v[80:95]}"(av[5]), "=&{v[96:111]}"(av[6]), "=&{v[112:127]}"(av[7]),
              [oc] "=s"(call), [l4] "+v"(l4), [wa] "+v"(wa), [ab] "+v"(ab), [pl] "=&v"(pl), [pd] "=&v"(pd), "+{v243}"(cp0), "+{v242}"(cp1), "+s"(wsw), "+s"(outp)
            : [vx] "v"(lane * 16 + wave * 4096), [vw] "v"(voffW), [kt] "v"(ktab), [vp] "v"(vp), [srda] "s"(srdA), [srdw] "s"(srdW),
              [ctr] "s"(ctr), [xp] "s"(xp), [pu] "s"(pu), [fl] "s"(fl), [tmo] "s"(tmo), [kh0] "s"(kh0), [ev] "s"(ev),
              [wv] "s"(wave), [r2] "s"(ro * 2048), [tp] "s"(tp), [tf] "s"(tf), [td] "s"(td),
              [s4] "s"(WAHEAD + ((wave >> 1) & 1)), [po] "s"(po), [vb] "s"((b & 1) ^ kf), [vo] "s"((u32)WS_FLAG + 8192 + b), [ld] "s"((b & ~4) * 8 + 1)
              , [dl] "v"(dl)
              , [xs] "s"(wsl + WS_X + (size_t)(b * 8 + wave) * 8192 + 4096 - wave * 4096)
              , [xf] "s"(wsl + WS_XF + (size_t)(b * 8 + wave) * 128)

            : "memory", "vcc", "scc",
              "v128", "v129", "v130", "v131", "v132", "v133", "v134", "v135", "v136", "v137", "v138", "v139", "v140", "v141", "v142", "v143", "v144", "v145", "v146", "v147", "v148", "v149", "v150", "v151", "v152", "v153", "v154", "v155", "v156", "v157", "v158", "v159", "v160", "v161", "v162", "v163", "v164", "v165", "v166", "v167", "v168", "v169", "v170", "v171", "v172", "v173", "v174", "v175", "v176", "v177", "v178", "v179", "v180", "v181", "v182", "v183", "v184", "v185", "v186", "v187", "v188", "v189", "v190", "v191", "v192", "v193", "v194", "v195", "v196", "v197", "v198", "v199", "v200", "v201", "v202", "v203", "v204", "v205", "v206", "v207", "v208", "v209", "v210", "v211", "v212", "v213", "v214", "v215", "v216", "v217", "v218", "v219", "v220", "v221", "v222", "v223", "v224", "v225", "v226", "v227", "v228", "v229", "v230", "v231", "v232", "v233", "v234", "v235", "v236", "v237", "v238", "v239", "v240", "v241",
              "s80", "s81", "s40", "s41", "s42", "s43", "s44", "s45", "s46", "s47", "s48", "s49", "s50", "s51", "s52", "s53", "s54", "s55", "s56", "s57", "s58", "s59");
    }
#define ACC(m, n, t) av[(m) * 2 + ((n) >> 2)][((n) & 3) * 4 + (t)]
        {
        const int lane = l4 >> 2;
        STAMP(14);
        EST(0); EST(1);

        av[0] = av[1]; av[1] = av[3]; av[2] = av[5]; av[3] = av[7];
        {
            char* xs = wsl + WS_X + (size_t)(b * 8 + wave) * 8192 + lane * 16;
            const char* xr = wsl + WS_X + (size_t)((b ^ 4) * 8 + wave) * 8192 + lane * 16;
            gu32* pf = (gu32*)(wsl + WS_XF + (size_t)((b ^ 4) * 8 + wave) * 128);
            u64 t0 = __builtin_amdgcn_s_memrealtime();
            for (;;) {
                u32 v = __hip_atomic_load(pf, __ATOMIC_RELAXED, __HIP_MEMORY_SCOPE_AGENT);
                if (__builtin_amdgcn_readfirstlane(v) >= call) break;
                if (__builtin_amdgcn_s_memrealtime() - t0 > DEADLINE_TICKS) {
                    __hip_atomic_store(tmo, call, __ATOMIC_RELAXED, __HIP_MEMORY_SCOPE_AGENT);
                    redu[16] = 1;
                    break;
                }
                __builtin_amdgcn_s_sleep(1);
            }
            STAMP(4);
            u32x4 v[8];
#pragma unroll
            for (int i = 0; i < 8; ++i)
                asm volatile("global_load_dwordx4 %0, %1, off offset:%2 sc1" : "=&v"(v[i]) : "v"(xr + (i >> 2) * 4096), "i"((i & 3) * 1024) : "memory");
            asm volatile("s_waitcnt vmcnt(0)" : "+v"(v[0]), "+v"(v[1]), "+v"(v[2]), "+v"(v[3]), "+v"(v[4]), "+v"(v[5]), "+v"(v[6]), "+v"(v[7]) :: "memory");
#pragma unroll
            for (int i = 0; i < 8; ++i) {
                const int j = i >> 2, n = (i & 3) * 2;
#pragma unroll
                for (int e = 0; e < 4; ++e) {
                    ACC(j, n + (e >> 1), (e & 1) * 2) += __builtin_bit_cast(float, v[i][e] << 16);
                    ACC(j, n + (e >> 1), (e & 1) * 2 + 1) += __builtin_bit_cast(float, v[i][e] & 0xffff0000u);
                }
            }
        }
        STAMP(5);
        EST(2);
        float m = 0.f;
#pragma unroll
        for (int j = 0; j < 2; ++j)
#pragma unroll
            for (int n = 0; n < 8; ++n)
                m = fmaxf(fmaxf(m, fmaxf(ACC(j, n, 0), ACC(j, n, 1))), fmaxf(ACC(j, n, 2), ACC(j, n, 3)));
        { const u32 mw = wave_maxu(__builtin_bit_cast(u32, m)); if (lane == 0) redu[wave] = mw; }
        BARRIER();
        STAMP(15);
        unsigned char* o = outp + (size_t)(wave * 64 + (lane & 15)) * NTOT + col0 + ro * cst + (lane >> 4) * 16;
#define OOFF(j, qh) ((size_t)((j) * 2 + (qh)) * 16 * NTOT)
        u32 amaxu = 0;
        {
            const u64 t0 = __builtin_amdgcn_s_memrealtime();
            if (wave == 2 && lane == 0) {
                u32 mm = 0;
#pragma unroll
                for (int i = 0; i < 8; ++i) mm = max(mm, redu[i]);
                if (redu[16] != 0) mm = 0xffffffffu;
                __hip_atomic_store((gu64*)(wsl + WS_GRAN + 4096) + (b & 7) * 32 + (b >> 3), ((u64)call << 32) | mm, __ATOMIC_RELAXED, __HIP_MEMORY_SCOPE_AGENT);
            }
#pragma unroll
        for (int j = 0; j < 4; ++j) {
#pragma unroll
            for (int e = 0; e < 16; ++e) { const float f = av[j][e]; av[j][e] = __builtin_bit_cast(float, max(__builtin_bit_cast(int, f), 0)); }
            asm volatile("" : "+v"(av[j]));
        }
            asm volatile("" : "+v"(o));
            if (wave == 0 || (wave == 1 && b < 8)) {
                char* wsr = P.ws[lane & 7];
                gu64* g = wave ? (gu64*)(wsl + WS_GRAN + 4096) + b * 32 + (lane & 31) : (gu64*)(wsl + WS_GRAN + (lane >> 3) * 128) + (lane & 7);
                u32 mu;
                for (;;) {
                    const u64 v = __hip_atomic_load(g, __ATOMIC_RELAXED, __HIP_MEMORY_SCOPE_SYSTEM);
                    mu = (u32)v;
                    if (__builtin_amdgcn_ballot_w64((u32)(v >> 32) != call) == 0) break;
                    if (__builtin_amdgcn_s_memrealtime() - t0 > DEADLINE_TICKS) {
                        __hip_atomic_store(tmo, call, __ATOMIC_RELAXED, __HIP_MEMORY_SCOPE_AGENT);
                        mu = 0xffffffffu;
                        break;
                    }
                    __builtin_amdgcn_s_sleep(GATH2);
                }
                mu = wave_maxu(mu);
                if (wave) {
                    if (lane < NR) __hip_atomic_store((gu64*)(wsr + WS_GRAN + rank * 128) + b, ((u64)call << 32) | mu, __ATOMIC_RELAXED, __HIP_MEMORY_SCOPE_SYSTEM);
                } else {
                    const float scale = __builtin_bit_cast(float, mu) / 448.0f;
                    redu[8] = __builtin_bit_cast(u32, scale == 0.f ? 0.f : 1.0f / scale);
                }
            }
        }
        EST(3);
        BARRIER();
#pragma unroll
        for (int i = 0; i < 8; ++i) amaxu = max(amaxu, redu[8 + i]);
        const bool bad = redu[16] != 0 || amaxu == 0xffffffffu;
        if (wave == 1 && lane == 0) __hip_atomic_store(ctr, call, __ATOMIC_RELAXED, __HIP_MEMORY_SCOPE_AGENT);
        STAMP(16);
        const float amax = __builtin_bit_cast(float, amaxu);
        const float scale = amax / 448.0f;
        const float inv = __builtin_bit_cast(float, (u32)redu[8]);
#pragma unroll
        for (int j = 0; j < 4; ++j) av[j] *= inv;
#pragma unroll
        for (int j = 0; j < 2; ++j)
#pragma unroll
            for (int qh = 0; qh < 2; ++qh) {
                u32x4 v;
#pragma unroll
                for (int t = 0; t < 4; ++t) {
                    int p = __builtin_amdgcn_cvt_pk_fp8_f32(ACC(j, qh * 4 + 0, t), ACC(j, qh * 4 + 1, t), 0, false);
                    v[t] = (u32)__builtin_amdgcn_cvt_pk_fp8_f32(ACC(j, qh * 4 + 2, t), ACC(j, qh * 4 + 3, t), p, true);
                }
                asm volatile("global_store_dwordx4 %0, %1, off " OUTP "\n\ts_nop 1" :: "v"(o + OOFF(j, qh)), "v"(v) : "memory");
            }
        asm volatile("a2a_code_end:");
        STAMP(17);
        STAMP(18);
        STAMPC(19, cstart);
        }
}

extern "C" void kernel_launch(int rank, int nranks, ncclComm_t comm,
                              hipStream_t stream,
                              void* const* d_in, const int* in_sizes, int n_in,
                              void* d_out, int out_size,
                              const int* in_dtypes, int out_dtype,
                              void* const* d_out_all,
                              void* const* d_ws_all, size_t ws_bytes) {
    if (nranks != NR || (unsigned)rank >= (unsigned)NR || n_in < 2 || ws_bytes < WS_END ||
        in_sizes[0] != KTOT * KC || in_sizes[1] != KTOT * NTOT || out_size != MLOC * NTOT) {
        fprintf(stderr, "kernel_launch: unexpected problem shape\n");
        return;
    }
    HIP_CHECK(hipFuncSetAttribute((const void*)a2a_gemm_kernel,
                                  hipFuncAttributeMaxDynamicSharedMemorySize, LDS_BYTES));
    Params p;
    p.x = (const float*)d_in[0];
    p.w = (const float*)d_in[1];
    p.out = (unsigned char*)d_out;
    for (int i = 0; i < NR; ++i) p.ws[i] = (char*)d_ws_all[(rank + i) & (NR - 1)];
    p.rank = rank;
    p.pad = 0;
    hipLaunchKernelGGL(a2a_gemm_kernel, dim3(NWG), dim3(NTHR), LDS_BYTES, stream, p);
    HIP_CHECK(hipGetLastError());
}
```
